```python
import numpy as np

kernel_name = "gpu_gemm_rs_m4096_k4096_n16384_bf16_q8"


ABSMAX_RTOL = 0.02

COMPUTE_DTYPE = "bfloat16"
EPILOGUE = "quant_int8"
IN_QUANT = None
OUT_QUANT = "int8"
IN_DTYPES = {"x": "bfloat16", "w_mat": "bfloat16"}
WORLD_SIZE = 8

def _to_bf16(x):
    u = np.ascontiguousarray(x, dtype=np.float32).view(np.uint32)
    return ((u + 0x7FFF + ((u >> 16) & 1)) & 0xFFFF0000).view(np.float32)

def setup_inputs():

    x = _to_bf16(np.random.randn(4096, 4096).astype(np.float32))
    w_mat = _to_bf16((np.random.randn(4096, 16384) / np.sqrt(4096)).astype(np.float32))
    return {"x": x, "w_mat": w_mat}

def _quant_dequant(y, amax):
    scale = amax / 127.0
    q = np.clip(np.round(y / scale), -127.0, 127.0)
    return (q * scale).astype(np.float32)
def _epilogue(y):
    return _quant_dequant(y, np.abs(y).max())

def reference(x, w_mat):
    y = (x @ w_mat).astype(np.float32)
    return _epilogue(y)
```

```cpp
#include <hip/hip_runtime.h>
#include <rccl/rccl.h>
#include <cstdio>
#include <cstdint>

#define HIP_CHECK(expr) do { hipError_t _e = (expr); if (_e != hipSuccess) { \
    fprintf(stderr, "HIP error %s at %s:%d\n", hipGetErrorString(_e), __FILE__, __LINE__); \
    return; } } while (0)

namespace {

#define LAS __attribute__((address_space(3)))
#define GAS __attribute__((address_space(1)))
typedef short bf16x8 __attribute__((ext_vector_type(8)));
typedef short s16x4 __attribute__((ext_vector_type(4)));
typedef float f32x4 __attribute__((ext_vector_type(4)));
typedef unsigned u32x4 __attribute__((ext_vector_type(4)));
typedef unsigned u32x2 __attribute__((ext_vector_type(2)));
typedef GAS unsigned gu32;
typedef GAS unsigned long long gu64;

constexpr int NR = 8;
constexpr int MM = 4096, NN = 16384;
constexpr int KL = 512;
constexpr int NL = NN / NR;
constexpr int ML = MM / NR;
constexpr int NSL = KL / 64;
constexpr int NRB = MM / 256;
constexpr int NCB = NL / 256;
constexpr int NKT = NR * NSL;
constexpr int CHUNK = 32768;
constexpr int NSEND = 128, NGEMM = 128, NBLK = NSEND + NGEMM, NTHR = 512;

constexpr size_t OFF_A = 0;
constexpr size_t OFF_B = (size_t)NR * NSL * NRB * CHUNK;
constexpr size_t OFF_C = OFF_B + (size_t)NR * NSL * NCB * CHUNK;
constexpr size_t C_FA = OFF_C;
constexpr size_t C_FB = C_FA + NRB * NR * 128;
constexpr size_t C_AMAX = C_FB + NCB * NR * 128;
constexpr size_t C_DONE = C_AMAX + NR * NGEMM * 8;
constexpr size_t C_CNT = C_DONE + NR * 128;
constexpr size_t C_TMO = C_CNT + NBLK * 64;
constexpr size_t C_STAMP = C_TMO + 128;
constexpr size_t WS_USED = C_STAMP + NBLK * 64 * 8;

constexpr int LDS_STAGE = 131072;
constexpr int LDS_CTL = LDS_STAGE;
constexpr int LDS_BYTES = LDS_STAGE + 1024;
constexpr unsigned long long DEADLINE = 40000000ull;

struct Params {
    const unsigned short* x;
    const unsigned short* w;
    unsigned char* ws[NR];
    signed char* out[NR];
    int rank;
    int pad;
};

__device__ __forceinline__ unsigned long long now() { return __builtin_amdgcn_s_memrealtime(); }
__device__ __forceinline__ unsigned ld_sys32(const void* p) { return __hip_atomic_load((gu32*)p, __ATOMIC_RELAXED, __HIP_MEMORY_SCOPE_SYSTEM); }
__device__ __forceinline__ void st_sys32(void* p, unsigned v) { __hip_atomic_store((gu32*)p, v, __ATOMIC_RELAXED, __HIP_MEMORY_SCOPE_SYSTEM); }
__device__ __forceinline__ unsigned ld_agent32(const void* p) { return __hip_atomic_load((gu32*)p, __ATOMIC_RELAXED, __HIP_MEMORY_SCOPE_AGENT); }
__device__ __forceinline__ void st_agent32(void* p, unsigned v) { __hip_atomic_store((gu32*)p, v, __ATOMIC_RELAXED, __HIP_MEMORY_SCOPE_AGENT); }
__device__ __forceinline__ void st_wt16(void* p, u32x4 v) {
    asm volatile("global_store_dwordx4 %0, %1, off sc0 sc1\n\ts_nop 1" :: "v"(p), "v"(v) : "memory");
}
#define WAIT_V(n) asm volatile("s_waitcnt vmcnt(" #n ")" ::: "memory")
#define WAIT_L(n) asm volatile("s_waitcnt lgkmcnt(" #n ")" ::: "memory")
#define BAR __builtin_amdgcn_s_barrier()
#define SCHED __builtin_amdgcn_sched_barrier(0)
#define CFENCE asm volatile("" ::: "memory")

#define STAMP(ws_own, idx) do {} while (0)

__device__ __forceinline__ int rot_of(int d) { return d == 0 ? 0 : ((d - 1) * 16) / 7; }

__device__ __forceinline__ bool give_up(unsigned long long t0, unsigned char* ws_own, unsigned call) {
    if (ld_agent32(ws_own + C_TMO) == call || now() - t0 > DEADLINE) { st_agent32(ws_own + C_TMO, call); return true; }
    return false;
}

__device__ __forceinline__ void sender(const Params& P, int g, unsigned call, LAS unsigned char* lds) {
    const int tid = threadIdx.x, wid = __builtin_amdgcn_readfirstlane(tid >> 6), lane = tid & 63;
    const int rank = P.rank;
    constexpr int SPL = NSEND / NR;
    const int d = g & 7, dest = (rank + d) & 7;
    unsigned char* wsd = P.ws[dest];
    const int rot = rot_of(d);
    unsigned char* own = P.ws[rank];
    STAMP(own, 0);
    for (int q = g >> 3; q < NSL * 24; q += SPL) {
        const int s = q / 24, u = q % 24;
        u32x4 v[4];
        unsigned char* dst; unsigned char* flag;
        if (u < NCB) {
            const int j = u;
            const unsigned char* src = (const unsigned char*)P.w + (size_t)(64 * s) * (NN * 2) + (size_t)(NL * dest + 256 * j) * 2;
#pragma unroll
            for (int it = 0; it < 4; ++it)
                v[it] = *(const GAS u32x4*)(src + (size_t)(it * 16 + (tid >> 5)) * (NN * 2) + (tid & 31) * 16);
#pragma unroll
            for (int it = 0; it < 4; ++it)
                *(LAS u32x4*)(lds + (it * 16 + (tid >> 5)) * 528 + (tid & 31) * 16) = v[it];
            __syncthreads();
            const int h = wid >> 2, wc = wid & 3, fg = lane >> 4, q4 = (lane >> 2) & 3, p4 = lane & 3;
#pragma unroll
            for (int ff = 0; ff < 4; ++ff) {
                const int n = ff >> 1, kf = ff & 1;
                LAS unsigned char* a = lds + (32 * kf + 8 * fg + q4) * 528 + (64 * wc + 16 * p4 + 4 * (2 * h + n)) * 2;
                typedef short v4i16_t __attribute__((ext_vector_type(4)));
                const v4i16_t lo = __builtin_amdgcn_ds_read_tr16_b64_v4i16((LAS v4i16_t*)a);
                const v4i16_t hi = __builtin_amdgcn_ds_read_tr16_b64_v4i16((LAS v4i16_t*)(a + 4 * 528));
                const u32x2 l2 = __builtin_bit_cast(u32x2, lo), h2 = __builtin_bit_cast(u32x2, hi);
                v[ff] = (u32x4){l2.x, l2.y, h2.x, h2.y};
            }
            dst = wsd + OFF_B + (size_t)((rank * NSL + s) * NCB + j) * CHUNK;
            flag = wsd + C_FB + (size_t)(j * NR + rank) * 128 + s * 4;
        } else {
            const int i = (u - NCB + rot) & 15;
            const int h = wid >> 2;
            const unsigned char* src = (const unsigned char*)P.x + (size_t)(256 * i + 128 * h + 32 * (wid & 3) + (lane & 15)) * (KL * 2) + (64 * s + 8 * (lane >> 4)) * 2;
#pragma unroll
            for (int ff = 0; ff < 4; ++ff)
                v[ff] = *(const GAS u32x4*)(src + (size_t)(16 * (ff >> 1)) * (KL * 2) + (ff & 1) * 64);
            dst = wsd + OFF_A + (size_t)((rank * NSL + s) * NRB + i) * CHUNK;
            flag = wsd + C_FA + (size_t)(i * NR + rank) * 128 + s * 4;
        }
        dst += (wid * 4) * 1024 + lane * 16;
#pragma unroll
        for (int ff = 0; ff < 4; ++ff) st_wt16(dst + ff * 1024, v[ff]);
        WAIT_V(0);
        __syncthreads();
        if (tid == 0) st_sys32(flag, call);
    }
    STAMP(own, 1);
    if (g == 0) {
        if (wid < 2) {
            const unsigned char* f = own + C_DONE + (size_t)(tid >> 4) * 128 + (tid & 15) * 4;
            const unsigned long long t0 = now();
            bool bad = false;
            while (!__all(ld_sys32(f) >= call)) {
                if (give_up(t0, own, call)) { bad = true; break; }
                __builtin_amdgcn_s_sleep(2);
            }
            if (bad && lane == 0) P.out[rank][rank * NL] = (signed char)-128;
        }
        STAMP(own, 2);
    }
}

__device__ __forceinline__ void gemm(const Params& P, int b, unsigned call, LAS unsigned char* lds) {
    const int tid = threadIdx.x, wid = __builtin_amdgcn_readfirstlane(tid >> 6), lane = tid & 63, wr = wid >> 2, wc = wid & 3;
    const int rank = P.rank;
    const int xcd = b & 7, slot = b >> 3;
    const int ti = 4 * (xcd >> 1) + (slot >> 2), tj = 4 * (xcd & 1) + (slot & 3);
    unsigned char* own = P.ws[rank];
    STAMP(own, 0);
    int dstar = 1;
#pragma unroll
    for (int d = 2; d < 8; ++d) if (rot_of(d) <= ti) dstar = d;
    dstar = __builtin_amdgcn_readfirstlane(dstar);
    auto ktile = [&](int t, int& p, int& s) {
        if (t < NSL) { p = rank; s = t; }
        else { const int u = t - NSL; s = (u * 37) >> 8; const int k = u - 7 * s; int d = dstar - k; if (d < 1) d += 7; p = (rank - d) & 7; }
    };
    auto kidx = [&](int t) -> unsigned { int p, s; ktile(t, p, s); return (unsigned)__builtin_amdgcn_readfirstlane(p * NSL + s); };
    const unsigned char* abase = own + OFF_A + (size_t)ti * CHUNK;
    const unsigned char* bbase = own + OFF_B + (size_t)tj * CHUNK;
    auto aptr = [&](int t) -> const unsigned char* { return abase + (size_t)kidx(t) * (NRB * CHUNK); };
    auto bptr = [&](int t) -> const unsigned char* { return bbase + (size_t)kidx(t) * (NCB * CHUNK); };

    LAS unsigned* ctl = (LAS unsigned*)(lds + LDS_CTL);
    const unsigned ctl_addr = (unsigned)(size_t)ctl;
    int ready = 0;
    auto ensure = [&](int need) {
        if (need >= ready) {
            if (wid == 0) {
                int t = need + (lane & 7); const bool act = lane < 16 && t < NKT; if (t >= NKT) t = NKT - 1;
                int p, s; ktile(t, p, s);
                const unsigned char* f = (lane & 8) ? own + C_FB + (size_t)(tj * NR + p) * 128 + s * 4 : own + C_FA + (size_t)(ti * NR + p) * 128 + s * 4;
                const unsigned long long t0 = now();
                int nr;
                for (;;) {
                    unsigned val = 0;
                    if (act) val = ld_sys32(f);
                    const unsigned long long m = __ballot(act && val >= call);
                    const unsigned both = (unsigned)(m & 0xffu) & (unsigned)((m >> 8) & 0xffu);
                    const int cnt = __builtin_ctz(~both);
                    if (cnt > 0) { nr = need + cnt; break; }
                    if (give_up(t0, own, call)) { nr = NKT; break; }
                    __builtin_amdgcn_s_sleep(2);
                }
                asm volatile("ds_write_b32 %0, %1\n\ts_waitcnt lgkmcnt(0)" :: "v"(ctl_addr), "v"(nr) : "memory");
            }
            CFENCE; BAR; CFENCE;
            int r;
            asm volatile("ds_read_b32 %0, %1\n\ts_waitcnt lgkmcnt(0)" : "=v"(r) : "v"(ctl_addr) : "memory");
            ready = __builtin_amdgcn_readfirstlane(r);
        }
    };

    f32x4 acc[2][2][4][2];
#pragma unroll
    for (int a = 0; a < 2; ++a)
#pragma unroll
        for (int c = 0; c < 2; ++c)
#pragma unroll
            for (int m = 0; m < 4; ++m)
#pragma unroll
                for (int n = 0; n < 2; ++n) acc[a][c][m][n] = (f32x4){0.f, 0.f, 0.f, 0.f};
    bf16x8 At[4][2], B0[2][2], B1[2][2];
    const unsigned voff = (unsigned)tid * 16u;
    const unsigned ldsw = (unsigned)wid * 1024u;
    const int aoff = wr * 8192 + lane * 16, boff = wc * 4096 + lane * 16;
    constexpr int HTB = 16384;
#define SA(b_, h_) (((b_) * 2 + (h_)) * HTB)
#define SB(b_, h_) ((4 + (b_) * 2 + (h_)) * HTB)
#define STAGE(bufoff, gbase) do { _Pragma("unroll") for (int _i = 0; _i < 2; ++_i) \
        __builtin_amdgcn_global_load_lds((const unsigned*)((const char*)(gbase) + voff + _i * 8192), (LAS unsigned*)(lds + (bufoff) + ldsw + _i * 8192), 16, 0, 17); } while (0)
#define LDA(dst, b_, h_) do { _Pragma("unroll") for (int m = 0; m < 4; ++m) _Pragma("unroll") for (int k = 0; k < 2; ++k) dst[m][k] = *(const LAS bf16x8*)(lds + SA(b_, h_) + aoff + m * 2048 + k * 1024); } while (0)
#define LDB(dst, b_, h_) do { _Pragma("unroll") for (int n = 0; n < 2; ++n) _Pragma("unroll") for (int k = 0; k < 2; ++k) dst[n][k] = *(const LAS bf16x8*)(lds + SB(b_, h_) + boff + n * 2048 + k * 1024); } while (0)
#define MMA(ai, bj, At_, Bt_) do { __builtin_amdgcn_s_setprio(1); _Pragma("unroll") for (int m = 0; m < 4; ++m) _Pragma("unroll") for (int n = 0; n < 2; ++n) _Pragma("unroll") for (int k = 0; k < 2; ++k) \
        acc[ai][bj][m][n] = __builtin_amdgcn_mfma_f32_16x16x32_bf16(Bt_[n][k], At_[m][k], acc[ai][bj][m][n], 0, 0, 0); __builtin_amdgcn_s_setprio(0); } while (0)

    ensure(0);
    {
        const unsigned char* a0 = aptr(0); const unsigned char* b0 = bptr(0);
        STAGE(SB(0, 0), b0); STAGE(SA(0, 0), a0); STAGE(SB(0, 1), b0 + HTB); STAGE(SA(0, 1), a0 + HTB);
    }
    ensure(1);
    if (wr == 1) BAR;
    WAIT_V(4); BAR;
    {
        const unsigned char* a1 = aptr(1); const unsigned char* b1 = bptr(1);
        STAGE(SB(1, 0), b1); STAGE(SA(1, 0), a1); STAGE(SB(1, 1), b1 + HTB);
    }
    WAIT_V(6); BAR;
    STAMP(own, 1);
    for (int t = 0; t < NKT; t += 2) {
        const bool last = (t == NKT - 2);
        const int t2 = last ? 0 : t + 2, t3 = last ? 1 : t + 3;
        const unsigned char* a1 = aptr(t + 1);
        const unsigned char* a2 = aptr(t2); const unsigned char* b2 = bptr(t2);
        const unsigned char* a3 = aptr(t3); const unsigned char* b3 = bptr(t3);
        LDB(B0, 0, 0); SCHED; LDA(At, 0, 0); STAGE(SA(1, 1), a1 + HTB);
        WAIT_L(8); BAR; WAIT_L(0); MMA(0, 0, At, B0); BAR; SCHED;
        ensure(t2);
        LDB(B1, 0, 1); STAGE(SB(0, 0), b2);
        BAR; WAIT_L(0); MMA(0, 1, At, B1); BAR;
        LDA(At, 0, 1); STAGE(SA(0, 0), a2);
        BAR; WAIT_L(0); MMA(1, 0, At, B0); BAR; SCHED;
        STAGE(SB(0, 1), b2 + HTB);
        WAIT_V(6); BAR; MMA(1, 1, At, B1); BAR;
        LDB(B0, 1, 0); SCHED; LDA(At, 1, 0); STAGE(SA(0, 1), a2 + HTB);
        WAIT_L(8); BAR; WAIT_L(0); MMA(0, 0, At, B0); BAR; SCHED;
        ensure(t3);
        LDB(B1, 1, 1); STAGE(SB(1, 0), b3);
        BAR; WAIT_L(0); MMA(0, 1, At, B1); BAR;
        LDA(At, 1, 1); STAGE(SA(1, 0), a3);
        BAR; WAIT_L(0); MMA(1, 0, At, B0); BAR; SCHED;
        STAGE(SB(1, 1), b3 + HTB);
        WAIT_V(6); BAR; MMA(1, 1, At, B1); BAR;
    }
    WAIT_V(0);
    if (wr == 0) BAR;
    BAR;
    STAMP(own, 2);

    int tz = threadIdx.x; asm volatile("" : "+v"(tz));
    const int lane2 = tz & 63, fr = lane2 & 15, fq = lane2 >> 4;
    float am = 0.f;
#pragma unroll
    for (int a = 0; a < 2; ++a)
#pragma unroll
        for (int c = 0; c < 2; ++c)
#pragma unroll
            for (int m = 0; m < 4; ++m)
#pragma unroll
                for (int n = 0; n < 2; ++n) {
                    const f32x4 v = acc[a][c][m][n];
                    am = fmaxf(am, fmaxf(fmaxf(fabsf(v[0]), fabsf(v[1])), fmaxf(fabsf(v[2]), fabsf(v[3]))));
                }
#pragma unroll
    for (int o = 32; o >= 1; o >>= 1) am = fmaxf(am, __shfl_xor(am, o));
    LAS float* red = (LAS float*)(lds + LDS_CTL + 64);
    if (lane2 == 0) red[wid] = am;
    __syncthreads();
    if (tz < NR) {
        float m8 = red[0];
#pragma unroll
        for (int k = 1; k < 8; ++k) m8 = fmaxf(m8, red[k]);
        const unsigned long long gran = ((unsigned long long)call << 32) | __float_as_uint(m8);
        __hip_atomic_store((gu64*)(P.ws[tz] + C_AMAX + (size_t)(rank * NGEMM + b) * 8), gran, __ATOMIC_RELAXED, __HIP_MEMORY_SCOPE_SYSTEM);
    }
    float gm;
    {
        gu64* g0 = (gu64*)(own + C_AMAX) + tz;
        const unsigned long long t0 = now();
        unsigned long long a0, a1;
        for (;;) {
            a0 = __hip_atomic_load(g0, __ATOMIC_RELAXED, __HIP_MEMORY_SCOPE_SYSTEM);
            a1 = __hip_atomic_load(g0 + 512, __ATOMIC_RELAXED, __HIP_MEMORY_SCOPE_SYSTEM);
            if (__all((unsigned)(a0 >> 32) >= call && (unsigned)(a1 >> 32) >= call)) break;
            if (give_up(t0, own, call)) break;
            __builtin_amdgcn_s_sleep(1);
        }
        gm = fmaxf(__uint_as_float((unsigned)a0), __uint_as_float((unsigned)a1));
#pragma unroll
        for (int o = 32; o >= 1; o >>= 1) gm = fmaxf(gm, __shfl_xor(gm, o));
        if (lane2 == 0) red[8 + wid] = gm;
        __syncthreads();
        gm = red[8];
#pragma unroll
        for (int k = 1; k < 8; ++k) gm = fmaxf(gm, red[8 + k]);
    }
    STAMP(own, 3);
    const bool poison = ld_agent32(own + C_TMO) == call;
    const float scale = gm / 127.0f;
    const float inv = scale > 0.f ? 1.0f / scale : 0.f;
    const int dest = ti >> 1;
    signed char* obase = P.out[dest] + (size_t)((ti & 1) * 256 + wr * 64 + fr) * NN + (size_t)rank * NL + tj * 256 + wc * 64 + fq * 16;
    constexpr float MAGIC = 12582912.0f;
#pragma unroll
    for (int a = 0; a < 2; ++a)
#pragma unroll
        for (int m = 0; m < 4; ++m) {
            u32x4 o;
#pragma unroll
            for (int c = 0; c < 2; ++c)
#pragma unroll
                for (int n = 0; n < 2; ++n) {
                    const f32x4 v = acc[a][c][m][n];
                    unsigned q[4];
#pragma unroll
                    for (int e = 0; e < 4; ++e)
                        q[e] = __float_as_uint(__builtin_amdgcn_fmed3f(__builtin_fmaf(v[e], inv, MAGIC), MAGIC - 127.0f, MAGIC + 127.0f));
                    const unsigned lo = __builtin_amdgcn_perm(q[1], q[0], 0x0c0c0400u), hi = __builtin_amdgcn_perm(q[3], q[2], 0x04000c0cu);
                    o[c * 2 + n] = lo | hi;
                }
            if (poison) o = (u32x4){0x80808080u, 0x80808080u, 0x80808080u, 0x80808080u};
            st_wt16(obase + (size_t)(a * 128 + m * 16) * NN, o);
        }
    WAIT_V(0);
    __syncthreads();
    if (tz == 0) st_sys32(P.ws[dest] + C_DONE + (size_t)rank * 128 + ((ti & 1) * 8 + tj) * 4, call);
    STAMP(own, 4);
#undef SA
#undef SB
#undef STAGE
#undef LDA
#undef LDB
#undef MMA
}

__global__ __launch_bounds__(NTHR, 2) void gemm_rs_q8_kernel(Params P) {
    extern __shared__ __attribute__((aligned(16))) unsigned char lds_raw[];
    LAS unsigned char* lds = (LAS unsigned char*)lds_raw;
    LAS unsigned* cw = (LAS unsigned*)(lds + LDS_CTL + 32);
    if (threadIdx.x == 0)
        *cw = __hip_atomic_fetch_add((gu32*)(P.ws[P.rank] + C_CNT + (size_t)blockIdx.x * 64), 1u, __ATOMIC_RELAXED, __HIP_MEMORY_SCOPE_AGENT) + 1u;
    __syncthreads();
    const unsigned call = __builtin_amdgcn_readfirstlane(*cw);
    if (blockIdx.x < NSEND) sender(P, blockIdx.x, call, lds);
    else gemm(P, blockIdx.x - NSEND, call, lds);
}

}

extern "C" void kernel_launch(int rank, int nranks, ncclComm_t comm,
                              hipStream_t stream,
                              void* const* d_in, const int* in_sizes, int n_in,
                              void* d_out, int out_size,
                              const int* in_dtypes, int out_dtype,
                              void* const* d_out_all,
                              void* const* d_ws_all, size_t ws_bytes) {
    (void)comm; (void)in_dtypes; (void)out_dtype; (void)d_out;
    if (nranks != NR || n_in != 2 || rank < 0 || rank >= NR || in_sizes[0] != MM * KL || in_sizes[1] != KL * NN ||
        out_size != ML * NN || ws_bytes < WS_USED) {
        fprintf(stderr, "kernel_launch: unexpected problem shape\n");
        return;
    }
    Params P;
    P.x = (const unsigned short*)d_in[0];
    P.w = (const unsigned short*)d_in[1];
    for (int p = 0; p < NR; ++p) { P.ws[p] = (unsigned char*)d_ws_all[p]; P.out[p] = (signed char*)d_out_all[p]; }
    P.rank = rank; P.pad = 0;
    HIP_CHECK(hipFuncSetAttribute((const void*)gemm_rs_q8_kernel, hipFuncAttributeMaxDynamicSharedMemorySize, LDS_BYTES));
    hipLaunchKernelGGL(gemm_rs_q8_kernel, dim3(NBLK), dim3(NTHR), LDS_BYTES, stream, P);
    HIP_CHECK(hipGetLastError());
}
```

```cpp
#include <hip/hip_runtime.h>
#include <rccl/rccl.h>
#include <cstdio>
#include <cstdint>

#define HIP_CHECK(expr) do { hipError_t _e = (expr); if (_e != hipSuccess) { \
    fprintf(stderr, "HIP error %s at %s:%d\n", hipGetErrorString(_e), __FILE__, __LINE__); \
    return; } } while (0)

namespace {

#define LAS __attribute__((address_space(3)))
#define GAS __attribute__((address_space(1)))
typedef short bf16x8 __attribute__((ext_vector_type(8)));
typedef short s16x4 __attribute__((ext_vector_type(4)));
typedef float f32x4 __attribute__((ext_vector_type(4)));
typedef unsigned u32x4 __attribute__((ext_vector_type(4)));
typedef unsigned u32x2 __attribute__((ext_vector_type(2)));
typedef GAS unsigned gu32;
typedef GAS unsigned long long gu64;

constexpr int NR = 8;
constexpr int MM = 4096, NN = 16384;
constexpr int KL = 512;
constexpr int NL = NN / NR;
constexpr int ML = MM / NR;
constexpr int NSL = KL / 64;
constexpr int NRB = MM / 256;
constexpr int NCB = NL / 256;
constexpr int NKT = NR * NSL;
constexpr int CHUNK = 32768;
constexpr int NSEND = 128, NGEMM = 128, NBLK = NSEND + NGEMM, NTHR = 512;

constexpr int NPC = NSL * 24 * 4;
constexpr int SLOT = 6144 + 256 + 128;
constexpr int EXC_CAP = 64;
constexpr size_t OFF_A = 0;
constexpr size_t OFF_B = (size_t)NR * NSL * NRB * CHUNK;
constexpr size_t OFF_P = OFF_B + (size_t)NR * NSL * NCB * CHUNK;
constexpr size_t OFF_C = OFF_P + (size_t)NR * NPC * SLOT;
constexpr size_t C_FA = OFF_C;
constexpr size_t C_FB = C_FA + NRB * NR * 128;
constexpr size_t C_AMAX = C_FB + NCB * NR * 128;
constexpr size_t C_DONE = C_AMAX + NR * NGEMM * 8;
constexpr size_t C_CNT = C_DONE + NR * 128;
constexpr size_t C_TICK = C_CNT + NBLK * 64;
constexpr size_t C_TMO = C_TICK + 2 * NR * 128;
constexpr size_t C_STAMP = C_TMO + 128;
constexpr int NSTAMP = 128;
constexpr size_t WS_USED = C_STAMP + NBLK * NSTAMP * 8;
static_assert(OFF_C % 128 == 0 && SLOT % 128 == 0, "alignment");

constexpr int LDS_STAGE = 131072;
constexpr int LDS_CTL = LDS_STAGE;
constexpr int LDS_BYTES = LDS_STAGE + 1024;
constexpr unsigned long long DEADLINE = 40000000ull;

struct Params {
    const unsigned short* x;
    const unsigned short* w;
    unsigned char* ws[NR];
    signed char* out[NR];
    int rank;
    int pad;
};

__device__ __forceinline__ unsigned long long now() { return __builtin_amdgcn_s_memrealtime(); }
__device__ __forceinline__ unsigned ld_sys32(const void* p) { return __hip_atomic_load((gu32*)p, __ATOMIC_RELAXED, __HIP_MEMORY_SCOPE_SYSTEM); }
__device__ __forceinline__ void st_sys32(void* p, unsigned v) { __hip_atomic_store((gu32*)p, v, __ATOMIC_RELAXED, __HIP_MEMORY_SCOPE_SYSTEM); }
__device__ __forceinline__ unsigned ld_agent32(const void* p) { return __hip_atomic_load((gu32*)p, __ATOMIC_RELAXED, __HIP_MEMORY_SCOPE_AGENT); }
__device__ __forceinline__ void st_agent32(void* p, unsigned v) { __hip_atomic_store((gu32*)p, v, __ATOMIC_RELAXED, __HIP_MEMORY_SCOPE_AGENT); }
__device__ __forceinline__ void st_wt16(void* p, u32x4 v) {
    asm volatile("global_store_dwordx4 %0, %1, off sc0 sc1\n\ts_nop 1" :: "v"(p), "v"(v) : "memory");
}
template <class T> __device__ __forceinline__ T* uni(T* p) {
    const unsigned long long v = (unsigned long long)p;
    const unsigned lo = __builtin_amdgcn_readfirstlane((unsigned)v), hi = __builtin_amdgcn_readfirstlane((unsigned)(v >> 32));
    return (T*)(((unsigned long long)hi << 32) | lo);
}
#define WAIT_V(n) asm volatile("s_waitcnt vmcnt(" #n ")" ::: "memory")
#define WAIT_L(n) asm volatile("s_waitcnt lgkmcnt(" #n ")" ::: "memory")
#define BAR __builtin_amdgcn_s_barrier()
#define SCHED __builtin_amdgcn_sched_barrier(0)
#define CFENCE asm volatile("" ::: "memory")

#define GAUX 17
#define STAMP(ws_own, idx) do {} while (0)
#define WSTAMP(ws_own, idx) do {} while (0)

__device__ __forceinline__ int rot_of(int d) { return d == 0 ? 0 : ((d - 1) * 16) / 7; }

__device__ __forceinline__ bool give_up(unsigned long long t0, unsigned char* ws_own, unsigned call) {
    if (ld_agent32(ws_own + C_TMO) == call || now() - t0 > DEADLINE) { st_agent32(ws_own + C_TMO, call); return true; }
    return false;
}

#define SEND_CUS 5
#define RAW_TAIL 0
constexpr int WAVE_LDS = 64 * 144 + 256 + 64;
typedef unsigned short u16x2 __attribute__((ext_vector_type(2)));
__device__ __forceinline__ u16x2 pk(unsigned v) { return __builtin_bit_cast(u16x2, v); }
__device__ __forceinline__ unsigned up(u16x2 v) { return __builtin_bit_cast(unsigned, v); }
template <int OFF> __device__ __forceinline__ void st_wt16o(void* p, u32x4 v) {
    asm volatile("global_store_dwordx4 %0, %1, off offset:%2 sc0 sc1\n\ts_nop 1" :: "v"(p), "v"(v), "i"(OFF) : "memory");
}
__device__ __forceinline__ void add_sys32(void* p) { (void)__hip_atomic_fetch_add((gu32*)p, 1u, __ATOMIC_RELAXED, __HIP_MEMORY_SCOPE_SYSTEM); }

struct PieceId { int isB, s, idx, pc; };
__device__ __forceinline__ PieceId piece_of(int q, int rot) {
    const int c = q >> 2, s = c / 24, u = c - 24 * s;
    PieceId id; id.isB = u < NCB; id.s = s; id.idx = id.isB ? u : ((u - NCB + rot) & 15); id.pc = q & 3; return id;
}
__device__ __forceinline__ unsigned char* image_of(unsigned char* ws, int src, const PieceId& id, int lane, int& stride4) {
    stride4 = id.isB ? 16384 : 4096;
    return id.isB ? ws + OFF_B + (size_t)((src * NSL + id.s) * NCB + id.idx) * CHUNK + id.pc * 4096 + lane * 16
                  : ws + OFF_A + (size_t)((src * NSL + id.s) * NRB + id.idx) * CHUNK + id.pc * 8192 + lane * 16;
}
__device__ __forceinline__ unsigned char* counter_of(unsigned char* ws, int src, const PieceId& id) {
    return (id.isB ? ws + C_FB : ws + C_FA) + (size_t)(id.idx * NR + src) * 128 + id.s * 4;
}
__device__ __forceinline__ void store_image(unsigned char* img, int stride4, const u32x4 (&v)[8]) {
    st_wt16o<0>(img, v[0]); st_wt16o<1024>(img, v[1]); st_wt16o<2048>(img, v[2]); st_wt16o<3072>(img, v[3]);
    unsigned char* i2 = img + stride4;
    st_wt16o<0>(i2, v[4]); st_wt16o<1024>(i2, v[5]); st_wt16o<2048>(i2, v[6]); st_wt16o<3072>(i2, v[7]);
}
__device__ __forceinline__ void issue_piece(const Params& P, int dest, const PieceId& id, int lane, u32x4 (&r)[8]) {
    if (id.isB) {
        const unsigned char* src = (const unsigned char*)P.w + (size_t)(64 * id.s + (lane >> 3)) * (NN * 2) + (size_t)(NL * dest + 256 * id.idx + 64 * id.pc + (lane & 7) * 8) * 2;
#pragma unroll
        for (int it = 0; it < 8; ++it) r[it] = *(const GAS u32x4*)(src + (size_t)(it * 8) * (NN * 2));
    } else {
        const unsigned char* src = (const unsigned char*)P.x + (size_t)(256 * id.idx + 64 * id.pc + (lane & 15)) * (KL * 2) + (64 * id.s + 8 * (lane >> 4)) * 2;
#pragma unroll
        for (int ff = 0; ff < 8; ++ff) r[ff] = *(const GAS u32x4*)(src + (size_t)(16 * (ff >> 1)) * (KL * 2) + (ff & 1) * 64);
    }
}
__device__ __forceinline__ void finish_piece(const PieceId& id, int lane, LAS unsigned char* wl, u32x4 (&v)[8]) {
    if (id.isB) {
#pragma unroll
        for (int it = 0; it < 8; ++it) *(LAS u32x4*)(wl + (it * 8 + (lane >> 3)) * 144 + (lane & 7) * 16) = v[it];
        WAIT_L(0);
        const int fg = lane >> 4, q4 = (lane >> 2) & 3, p4 = lane & 3;
        LAS unsigned char* a0 = wl + (8 * fg + q4) * 144 + p4 * 32;
#pragma unroll
        for (int ff = 0; ff < 8; ++ff) {
            const int hn = ff >> 1, kf = ff & 1;
            LAS unsigned char* a = a0 + (32 * kf) * 144 + hn * 8;
            typedef short v4i16_t __attribute__((ext_vector_type(4)));
            const v4i16_t lo = __builtin_amdgcn_ds_read_tr16_b64_v4i16((LAS v4i16_t*)a);
            const v4i16_t hi = __builtin_amdgcn_ds_read_tr16_b64_v4i16((LAS v4i16_t*)(a + 4 * 144));
            const u32x2 l2 = __builtin_bit_cast(u32x2, lo), h2 = __builtin_bit_cast(u32x2, hi);
            v[ff] = (u32x4){l2.x, l2.y, h2.x, h2.y};
        }
        WAIT_L(0);
    }
}

__device__ __forceinline__ void comm(const Params& P, int b, unsigned call, LAS unsigned char* lds) {
    const int tid = threadIdx.x, wid = __builtin_amdgcn_readfirstlane(tid >> 6), lane = tid & 63;
    const int rank = P.rank;
    const int xcd = b & 7, slot = b >> 3;
    unsigned char* own = P.ws[rank];
    LAS unsigned char* wl = lds + wid * WAVE_LDS;
    STAMP(own, 0);
    if (xcd == 0 || slot < SEND_CUS) {
        const int d = xcd, dest = (rank + d) & 7, rot = rot_of(d);
        const int nw = d == 0 ? 128 : SEND_CUS * 8;
        unsigned char* wsd = P.ws[dest];
        gu32* tick = (gu32*)(own + C_TICK + (size_t)d * 128);
        const unsigned tbase = (call - 1u) * (unsigned)NPC - (unsigned)nw;
        u32x4 nxt[8];
        int q = slot * 8 + wid;
        issue_piece(P, dest, piece_of(q, rot), lane, nxt);
        unsigned tk = 0;
        if (lane == 0) tk = __hip_atomic_fetch_add(tick, 1u, __ATOMIC_RELAXED, __HIP_MEMORY_SCOPE_AGENT);
        unsigned char* pub = nullptr; unsigned long long pubv = 0; bool pubdirect = false;
        for (;;) {
            const PieceId id = piece_of(q, rot);
            u32x4 v[8];
            asm volatile("s_waitcnt vmcnt(0)" : "+v"(nxt[0]), "+v"(nxt[1]), "+v"(nxt[2]), "+v"(nxt[3]), "+v"(nxt[4]), "+v"(nxt[5]), "+v"(nxt[6]), "+v"(nxt[7]), "+v"(tk) :: "memory");
            if (pub && lane == 0) {
                if (pubdirect) add_sys32(pub);
                else __hip_atomic_store((gu64*)pub, pubv, __ATOMIC_RELAXED, __HIP_MEMORY_SCOPE_SYSTEM);
            }
#pragma unroll
            for (int ff = 0; ff < 8; ++ff) v[ff] = nxt[ff];
            const unsigned qn = (unsigned)__builtin_amdgcn_readfirstlane(tk) - tbase;
            const bool more = qn < (unsigned)NPC;
            if (more) {
                issue_piece(P, dest, piece_of((int)qn, rot), lane, nxt);
                if (lane == 0) tk = __hip_atomic_fetch_add(tick, 1u, __ATOMIC_RELAXED, __HIP_MEMORY_SCOPE_AGENT);
            }
            finish_piece(id, lane, wl, v);
            bool raw = d == 0 || q >= NPC - RAW_TAIL;
            const bool direct = raw;
            unsigned meta = 0x100u;
            unsigned char* sl = wsd + OFF_P + (size_t)(rank * NPC + q) * SLOT;
            if (!raw) {
                u16x2 mx = pk(0u), mn = pk(0x00ff00ffu);
#pragma unroll
                for (int ff = 0; ff < 8; ++ff)
#pragma unroll
                    for (int i = 0; i < 4; ++i) { const u16x2 ex = pk((v[ff][i] >> 7) & 0x00ff00ffu); mx = __builtin_elementwise_max(mx, ex); mn = __builtin_elementwise_min(mn, ex); }
                unsigned hi = max((unsigned)mx.x, (unsigned)mx.y), lo = min((unsigned)mn.x, (unsigned)mn.y);
                const unsigned lane_lo = lo;
#pragma unroll
                for (int o = 32; o >= 1; o >>= 1) { hi = max(hi, (unsigned)__shfl_xor((int)hi, o)); lo = min(lo, (unsigned)__shfl_xor((int)lo, o)); }
                hi = __builtin_amdgcn_readfirstlane(hi); lo = __builtin_amdgcn_readfirstlane(lo);
                const unsigned base = hi > 15u ? hi - 15u : 0u, base2 = base * 0x00010001u;
                unsigned nexc = 0;
                if (lo < base) {
                    LAS unsigned* lst = (LAS unsigned*)(wl + 64 * 144);
                    LAS unsigned* cnt = lst + EXC_CAP;
                    if (lane == 0) *cnt = 0u;
                    WAIT_L(0);
                    if (lane_lo < base) {
#pragma unroll
                        for (int ff = 0; ff < 8; ++ff) {
                            unsigned m = 0;
#pragma unroll
                            for (int i = 0; i < 4; ++i) {
                                const u16x2 ex = pk((v[ff][i] >> 7) & 0x00ff00ffu);
                                m |= up(__builtin_elementwise_min(__builtin_elementwise_sub_sat(pk(base2), ex), pk(0x00010001u))) << i;
                            }
                            while (m) {
                                const int bit = __builtin_ctz(m); m &= m - 1;
                                const int i = bit & 3, half = bit >> 4;
                                const unsigned dw = i == 0 ? v[ff][0] : i == 1 ? v[ff][1] : i == 2 ? v[ff][2] : v[ff][3];
                                const unsigned val = half ? dw >> 16 : dw & 0xffffu;
                                const unsigned pos = __hip_atomic_fetch_add(cnt, 1u, __ATOMIC_RELAXED, __HIP_MEMORY_SCOPE_WORKGROUP);
                                if (pos < EXC_CAP) lst[pos] = ((unsigned)((ff * 64 + lane) * 8 + 2 * i + half) << 16) | val;
                            }
                        }
                    }
                    WAIT_L(0);
                    nexc = __builtin_amdgcn_readfirstlane(*(volatile LAS unsigned*)cnt);
                    if (nexc > EXC_CAP) raw = true;
                    else {
                        const unsigned e = *(volatile LAS unsigned*)(lst + lane);
                        asm volatile("global_store_dword %0, %1, off sc0 sc1\n\ts_nop 1" :: "v"(sl + 6144 + lane * 4), "v"(e) : "memory");
                    }
                }
                if (!raw) {
                    u32x4 pkd[6];
#pragma unroll
                    for (int ff = 0; ff < 8; ++ff) {
                        unsigned n = 0, t[4];
#pragma unroll
                        for (int i = 0; i < 4; ++i) {
                            const unsigned dw = v[ff][i];
                            n |= up(__builtin_elementwise_sub_sat(pk((dw >> 7) & 0x00ff00ffu), pk(base2))) << (4 * i);
                            t[i] = (dw & 0x007f007fu) | ((dw >> 8) & ~0x007f007fu);
                        }
                        pkd[ff >> 1][(ff & 1) * 2] = __builtin_amdgcn_perm(t[1], t[0], 0x06040200u);
                        pkd[ff >> 1][(ff & 1) * 2 + 1] = __builtin_amdgcn_perm(t[3], t[2], 0x06040200u);
                        pkd[4 + (ff >> 2)][ff & 3] = n;
                    }
                    unsigned char* a = sl + lane * 16;
                    st_wt16o<0>(a, pkd[0]); st_wt16o<1024>(a, pkd[1]); st_wt16o<2048>(a, pkd[2]); st_wt16o<3072>(a, pkd[3]);
                    st_wt16o<0>(a + 4096, pkd[4]); st_wt16o<1024>(a + 4096, pkd[5]);
                    meta = base | (nexc << 16);
                }
            }
            if (raw) { int st4; unsigned char* img = image_of(wsd, rank, id, lane, st4); store_image(img, st4, v); }
            pubdirect = direct;
            pub = direct ? counter_of(wsd, rank, id) : sl + 6144 + 256;
            pubv = ((unsigned long long)call << 32) | meta;
            if (!more) break;
            q = (int)qn;
        }
        WAIT_V(0);
        if (pub && lane == 0) {
            if (pubdirect) add_sys32(pub);
            else __hip_atomic_store((gu64*)pub, pubv, __ATOMIC_RELAXED, __HIP_MEMORY_SCOPE_SYSTEM);
        }
        STAMP(own, 1); WSTAMP(own, 8);
    } else {
        const int d = xcd, src = (rank - d) & 7, rot = rot_of(d);
        const int nw = (16 - SEND_CUS) * 8;
        constexpr int NUP = NPC - RAW_TAIL;
        gu32* tick = (gu32*)(own + C_TICK + (size_t)(NR + d) * 128);
        const unsigned tbase = (call - 1u) * (unsigned)NUP - (unsigned)nw;
        unsigned tk = 0;
        for (int q = (slot - SEND_CUS) * 8 + wid; q < NUP;) {
            if (lane == 0) tk = __hip_atomic_fetch_add(tick, 1u, __ATOMIC_RELAXED, __HIP_MEMORY_SCOPE_AGENT);
            const PieceId id = piece_of(q, rot);
            const unsigned char* sl = own + OFF_P + (size_t)(src * NPC + q) * SLOT;
            unsigned meta;
            {
                const unsigned long long t0 = now();
                for (;;) {
                    const unsigned long long g = __hip_atomic_load((gu64*)(sl + 6144 + 256), __ATOMIC_RELAXED, __HIP_MEMORY_SCOPE_SYSTEM);
                    const unsigned tag = __builtin_amdgcn_readfirstlane((unsigned)(g >> 32));
                    meta = __builtin_amdgcn_readfirstlane((unsigned)g);
                    if (tag >= call) break;
                    if (give_up(t0, own, call)) { meta = 0x100u; break; }
                    __builtin_amdgcn_s_sleep(2);
                }
            }
            if (!(meta & 0x100u)) {
                u32x4 k0, k1, k2, k3, k4, k5; unsigned exc;
                const unsigned char* a = sl + lane * 16;
                asm volatile("global_load_dwordx4 %0, %7, off sc0 sc1\n\t"
                             "global_load_dwordx4 %1, %7, off offset:1024 sc0 sc1\n\t"
                             "global_load_dwordx4 %2, %7, off offset:2048 sc0 sc1\n\t"
                             "global_load_dwordx4 %3, %7, off offset:3072 sc0 sc1\n\t"
                             "global_load_dwordx4 %4, %8, off sc0 sc1\n\t"
                             "global_load_dwordx4 %5, %8, off offset:1024 sc0 sc1\n\t"
                             "global_load_dword %6, %9, off sc0 sc1\n\t"
                             "s_waitcnt vmcnt(0)"
                             : "=&v"(k0), "=&v"(k1), "=&v"(k2), "=&v"(k3), "=&v"(k4), "=&v"(k5), "=&v"(exc)
                             : "v"(a), "v"(a + 4096), "v"(sl + 6144 + lane * 4) : "memory");
                const u32x4 pkd[6] = {k0, k1, k2, k3, k4, k5};
                const unsigned base7 = ((meta & 0xffu) << 7) * 0x00010001u;
                u32x4 v[8];
#pragma unroll
                for (int ff = 0; ff < 8; ++ff) {
                    const unsigned n = pkd[4 + (ff >> 2)][ff & 3];
#pragma unroll
                    for (int i = 0; i < 4; ++i) {
                        const unsigned sm = pkd[ff >> 1][(ff & 1) * 2 + (i >> 1)];
                        const unsigned u = __builtin_amdgcn_perm(sm, sm, (i & 1) ? 0x03030202u : 0x01010000u);
                        const unsigned e = (((n >> (4 * i)) & 0x000f000fu) << 7) + base7;
                        v[ff][i] = (u & 0x807f807fu) | e;
                    }
                }
                int st4; unsigned char* img = image_of(own, src, id, lane, st4);
                store_image(img, st4, v);
                const unsigned nexc = meta >> 16;
                if (nexc) {
                    WAIT_V(0);
                    if ((unsigned)lane < nexc) {
                        const unsigned el = (exc >> 16) & 4095u, ff = el >> 9;
                        unsigned char* dst = img - lane * 16 + (ff >> 2) * st4 + (ff & 3) * 1024 + (el & 511u) * 2;
                        asm volatile("global_store_short %0, %1, off sc0 sc1\n\ts_nop 1" :: "v"(dst), "v"(exc) : "memory");
                    }
                }
                WAIT_V(0);
            }
            if (lane == 0) add_sys32(counter_of(own, src, id));
            const unsigned qn = (unsigned)__builtin_amdgcn_readfirstlane(tk) - tbase;
            q = qn < (unsigned)NUP ? (int)qn : NUP;
        }
        STAMP(own, 1); WSTAMP(own, 8);
    }
    if (b == 0) {
        if (wid < 2) {
            const unsigned char* f = own + C_DONE + (size_t)(tid >> 4) * 128 + (tid & 15) * 4;
            const unsigned long long t0 = now();
            bool bad = false;
            while (!__all(ld_sys32(f) >= call)) {
                if (give_up(t0, own, call)) { bad = true; break; }
                __builtin_amdgcn_s_sleep(2);
            }
            if (bad && lane == 0) P.out[rank][rank * NL] = (signed char)-128;
        }
        STAMP(own, 2);
    }
}

__device__ __forceinline__ void gemm(const Params& P, int b, unsigned call, LAS unsigned char* lds) {
    const int tid = threadIdx.x, wid = __builtin_amdgcn_readfirstlane(tid >> 6), lane = tid & 63, wr = wid >> 2, wc = wid & 3;
    const int rank = P.rank;
    const int xcd = b & 7, slot = b >> 3;
    const int ti = 2 * ((rank + xcd) & 7) + (slot >> 3), tj = slot & 7;
    unsigned char* own = P.ws[rank];
    STAMP(own, 0);
    const unsigned char* abase = uni(own + OFF_A + (size_t)ti * CHUNK);
    const unsigned char* bbase = uni(own + OFF_B + (size_t)tj * CHUNK);
    auto aptr = [&](int k) -> const unsigned char* { return abase + ((size_t)(unsigned)k << 19); };
    auto bptr = [&](int k) -> const unsigned char* { return bbase + ((size_t)(unsigned)k << 18); };
    static_assert(NRB * CHUNK == (1 << 19) && NCB * CHUNK == (1 << 18), "chunk strides");

    LAS unsigned* ctl = (LAS unsigned*)(lds + LDS_CTL);
    const unsigned ctl_addr = (unsigned)(size_t)ctl;
    const unsigned char* fbaseA = uni(own + C_FA + (size_t)ti * NR * 128);
    const unsigned char* fbaseB = uni(own + C_FB + (size_t)tj * NR * 128);
    const unsigned foff = (unsigned)(lane >> 3) * 128u + (unsigned)(lane & 7) * 4u;
    const unsigned target = 4u * call;
    unsigned long long known = 0, taken = 0;
    auto pick = [&]() -> int {
        if ((known & ~taken) == 0) {
            if (taken == ~0ull) return 0;
            if (wid == 0) {
                const unsigned long long t0 = now();
                unsigned long long m;
                for (;;) {
                    unsigned va, vb;
                    asm volatile("global_load_dword %0, %2, %3 sc0 sc1\n\tglobal_load_dword %1, %2, %4 sc0 sc1\n\ts_waitcnt vmcnt(0)"
                                 : "=&v"(va), "=&v"(vb) : "v"(foff), "s"(fbaseA), "s"(fbaseB) : "memory");
                    m = __ballot(va >= target && vb >= target);
                    if (m & ~taken) break;
                    if (give_up(t0, own, call)) { m = ~0ull; break; }
                    __builtin_amdgcn_s_sleep(1);
                }
                const u32x2 mv = {(unsigned)m, (unsigned)(m >> 32)};
                asm volatile("ds_write_b64 %0, %1\n\ts_waitcnt lgkmcnt(0)" :: "v"(ctl_addr), "v"(mv) : "memory");
            }
            CFENCE; BAR; CFENCE;
            u32x2 r;
            asm volatile("ds_read_b64 %0, %1\n\ts_waitcnt lgkmcnt(0)" : "=v"(r) : "v"(ctl_addr) : "memory");
            known |= ((unsigned long long)(unsigned)__builtin_amdgcn_readfirstlane(r.y) << 32) | (unsigned)__builtin_amdgcn_readfirstlane(r.x);
        }
        const int k = __builtin_ctzll(known & ~taken);
        taken |= 1ull << k;
        return k;
    };

    f32x4 acc[2][2][4][2];
#pragma unroll
    for (int a = 0; a < 2; ++a)
#pragma unroll
        for (int c = 0; c < 2; ++c)
#pragma unroll
            for (int m = 0; m < 4; ++m)
#pragma unroll
                for (int n = 0; n < 2; ++n) acc[a][c][m][n] = (f32x4){0.f, 0.f, 0.f, 0.f};
    bf16x8 At[4][2], B0[2][2], B1[2][2];
    unsigned voff[2] = {(unsigned)tid * 16u, (unsigned)tid * 16u + 8192u};
    asm volatile("" : "+v"(voff[0]), "+v"(voff[1]));
    const unsigned ldsw = (unsigned)wid * 1024u;
    const int aoff = wr * 8192 + lane * 16, boff = wc * 4096 + lane * 16;
    constexpr int HTB = 16384;
#define SA(b_, h_) (((b_) * 2 + (h_)) * HTB)
#define SB(b_, h_) ((4 + (b_) * 2 + (h_)) * HTB)
#define STAGE(bufoff, gbase) do { _Pragma("unroll") for (int _i = 0; _i < 2; ++_i) \
        __builtin_amdgcn_global_load_lds((const unsigned*)((const char*)(gbase) + voff[_i]), (LAS unsigned*)(lds + (bufoff) + ldsw + _i * 8192), 16, 0, GAUX); } while (0)
#define LDA(dst, b_, h_) do { _Pragma("unroll") for (int m = 0; m < 4; ++m) _Pragma("unroll") for (int k = 0; k < 2; ++k) dst[m][k] = *(const LAS bf16x8*)(lds + SA(b_, h_) + aoff + m * 2048 + k * 1024); } while (0)
#define LDB(dst, b_, h_) do { _Pragma("unroll") for (int n = 0; n < 2; ++n) _Pragma("unroll") for (int k = 0; k < 2; ++k) dst[n][k] = *(const LAS bf16x8*)(lds + SB(b_, h_) + boff + n * 2048 + k * 1024); } while (0)
#define MMA(ai, bj, At_, Bt_) do { __builtin_amdgcn_s_setprio(1); _Pragma("unroll") for (int m = 0; m < 4; ++m) _Pragma("unroll") for (int n = 0; n < 2; ++n) _Pragma("unroll") for (int k = 0; k < 2; ++k) \
        acc[ai][bj][m][n] = __builtin_amdgcn_mfma_f32_16x16x32_bf16(Bt_[n][k], At_[m][k], acc[ai][bj][m][n], 0, 0, 0); __builtin_amdgcn_s_setprio(0); } while (0)

    {
        const int k0 = pick();
        const unsigned char* a0 = aptr(k0); const unsigned char* b0 = bptr(k0);
        STAGE(SB(0, 0), b0); STAGE(SA(0, 0), a0); STAGE(SB(0, 1), b0 + HTB); STAGE(SA(0, 1), a0 + HTB);
    }
    int k1 = pick();
    if (wr == 1) BAR;
    WAIT_V(4); BAR;
    {
        const unsigned char* a1 = aptr(k1); const unsigned char* b1 = bptr(k1);
        STAGE(SB(1, 0), b1); STAGE(SA(1, 0), a1); STAGE(SB(1, 1), b1 + HTB);
    }
    WAIT_V(6); BAR;
    STAMP(own, 1);
    for (int t = 0; t < NKT; t += 2) {
        const unsigned char* a1 = aptr(k1);
        unsigned pa = 0, pb = 0;
        const bool look = wid == 0 && known != ~0ull;
        if (look) asm volatile("global_load_dword %0, %2, %3 sc0 sc1\n\tglobal_load_dword %1, %2, %4 sc0 sc1" : "=&v"(pa), "=&v"(pb) : "v"(foff), "s"(fbaseA), "s"(fbaseB) : "memory");
        LDB(B0, 0, 0); SCHED; LDA(At, 0, 0); STAGE(SA(1, 1), a1 + HTB);
        WAIT_L(8); BAR; WAIT_L(0); MMA(0, 0, At, B0); BAR; SCHED;
        const int k2 = pick();
        const unsigned char* a2 = aptr(k2); const unsigned char* b2 = bptr(k2);
        LDB(B1, 0, 1); STAGE(SB(0, 0), b2);
        BAR; WAIT_L(0); MMA(0, 1, At, B1); BAR;
        LDA(At, 0, 1); STAGE(SA(0, 0), a2);
        BAR; WAIT_L(0); MMA(1, 0, At, B0); BAR; SCHED;
        STAGE(SB(0, 1), b2 + HTB);
        WAIT_V(6);
        if (wid == 0) {
            unsigned long long m = known;
            if (look) { asm volatile("" : "+v"(pa), "+v"(pb)); m |= __ballot(pa >= target && pb >= target); }
            const u32x2 mv = {(unsigned)m, (unsigned)(m >> 32)};
            asm volatile("ds_write_b64 %0, %1 offset:8\n\ts_waitcnt lgkmcnt(0)" :: "v"(ctl_addr), "v"(mv) : "memory");
        }
        BAR; MMA(1, 1, At, B1); BAR;
        u32x2 rnew;
        asm volatile("ds_read_b64 %0, %1 offset:8" : "=v"(rnew) : "v"(ctl_addr) : "memory");
        LDB(B0, 1, 0); SCHED; LDA(At, 1, 0); STAGE(SA(0, 1), a2 + HTB);
        WAIT_L(8); BAR; WAIT_L(0);
        asm volatile("" : "+v"(rnew));
        known |= ((unsigned long long)(unsigned)__builtin_amdgcn_readfirstlane(rnew.y) << 32) | (unsigned)__builtin_amdgcn_readfirstlane(rnew.x);
        MMA(0, 0, At, B0); BAR; SCHED;
        const int k3 = pick();
        const unsigned char* a3 = aptr(k3); const unsigned char* b3 = bptr(k3);
        LDB(B1, 1, 1); STAGE(SB(1, 0), b3);
        BAR; WAIT_L(0); MMA(0, 1, At, B1); BAR;
        LDA(At, 1, 1); STAGE(SA(1, 0), a3);
        BAR; WAIT_L(0); MMA(1, 0, At, B0); BAR; SCHED;
        STAGE(SB(1, 1), b3 + HTB);
        WAIT_V(6); BAR; MMA(1, 1, At, B1); BAR;
        k1 = k3;
    }
    WAIT_V(0);
    if (wr == 0) BAR;
    BAR;
    STAMP(own, 2);

    int tz = threadIdx.x; asm volatile("" : "+v"(tz));
    const int lane2 = tz & 63, fr = lane2 & 15, fq = lane2 >> 4;
    float am = 0.f;
#pragma unroll
    for (int a = 0; a < 2; ++a)
#pragma unroll
        for (int c = 0; c < 2; ++c)
#pragma unroll
            for (int m = 0; m < 4; ++m)
#pragma unroll
                for (int n = 0; n < 2; ++n) {
                    const f32x4 v = acc[a][c][m][n];
                    am = fmaxf(am, fmaxf(fmaxf(fabsf(v[0]), fabsf(v[1])), fmaxf(fabsf(v[2]), fabsf(v[3]))));
                }
#pragma unroll
    for (int o = 32; o >= 1; o >>= 1) am = fmaxf(am, __shfl_xor(am, o));
    LAS float* red = (LAS float*)(lds + LDS_CTL + 64);
    if (lane2 == 0) red[wid] = am;
    __syncthreads();
    if (tz < NR) {
        float m8 = red[0];
#pragma unroll
        for (int k = 1; k < 8; ++k) m8 = fmaxf(m8, red[k]);
        const unsigned long long gran = ((unsigned long long)call << 32) | __float_as_uint(m8);
        __hip_atomic_store((gu64*)(P.ws[tz] + C_AMAX + (size_t)(rank * NGEMM + b) * 8), gran, __ATOMIC_RELAXED, __HIP_MEMORY_SCOPE_SYSTEM);
    }
    STAMP(own, 10);
    float gm;
    {
        gu64* g0 = (gu64*)(own + C_AMAX) + tz;
        const unsigned long long t0 = now();
        unsigned long long a0, a1;
        for (;;) {
            a0 = __hip_atomic_load(g0, __ATOMIC_RELAXED, __HIP_MEMORY_SCOPE_SYSTEM);
            a1 = __hip_atomic_load(g0 + 512, __ATOMIC_RELAXED, __HIP_MEMORY_SCOPE_SYSTEM);
            if (__all((unsigned)(a0 >> 32) >= call && (unsigned)(a1 >> 32) >= call)) break;
            if (give_up(t0, own, call)) break;
            __builtin_amdgcn_s_sleep(1);
        }
        gm = fmaxf(__uint_as_float((unsigned)a0), __uint_as_float((unsigned)a1));
#pragma unroll
        for (int o = 32; o >= 1; o >>= 1) gm = fmaxf(gm, __shfl_xor(gm, o));
        if (lane2 == 0) red[8 + wid] = gm;
        __syncthreads();
        gm = red[8];
#pragma unroll
        for (int k = 1; k < 8; ++k) gm = fmaxf(gm, red[8 + k]);
    }
    STAMP(own, 3);
    const bool poison = ld_agent32(own + C_TMO) == call;
    const float scale = gm / 127.0f;
    const float inv = scale > 0.f ? 1.0f / scale : 0.f;
    const int dest = ti >> 1;
    signed char* obase = P.out[dest] + (size_t)((ti & 1) * 256 + wr * 64 + fr) * NN + (size_t)rank * NL + tj * 256 + wc * 64 + fq * 16;
    constexpr float MAGIC = 12582912.0f;
#pragma unroll
    for (int a = 0; a < 2; ++a)
#pragma unroll
        for (int m = 0; m < 4; ++m) {
            u32x4 o;
#pragma unroll
            for (int c = 0; c < 2; ++c)
#pragma unroll
                for (int n = 0; n < 2; ++n) {
                    const f32x4 v = acc[a][c][m][n];
                    unsigned q[4];
#pragma unroll
                    for (int e = 0; e < 4; ++e)
                        q[e] = __float_as_uint(__builtin_amdgcn_fmed3f(__builtin_fmaf(v[e], inv, MAGIC), MAGIC - 127.0f, MAGIC + 127.0f));
                    const unsigned lo = __builtin_amdgcn_perm(q[1], q[0], 0x0c0c0400u), hi = __builtin_amdgcn_perm(q[3], q[2], 0x04000c0cu);
                    o[c * 2 + n] = lo | hi;
                }
            if (poison) o = (u32x4){0x80808080u, 0x80808080u, 0x80808080u, 0x80808080u};
            st_wt16(obase + (size_t)(a * 128 + m * 16) * NN, o);
        }
    WAIT_V(0);
    __syncthreads();
    if (tz == 0) st_sys32(P.ws[dest] + C_DONE + (size_t)rank * 128 + ((ti & 1) * 8 + tj) * 4, call);
    STAMP(own, 4);
#undef SA
#undef SB
#undef STAGE
#undef LDA
#undef LDB
#undef MMA
}

__global__ __launch_bounds__(NTHR, 2) void gemm_rs_q8_kernel(Params P) {
    extern __shared__ __attribute__((aligned(16))) unsigned char lds_raw[];
    LAS unsigned char* lds = (LAS unsigned char*)lds_raw;
    LAS unsigned* cw = (LAS unsigned*)(lds + LDS_CTL + 32);
    if (threadIdx.x == 0)
        *cw = __hip_atomic_fetch_add((gu32*)(P.ws[P.rank] + C_CNT + (size_t)blockIdx.x * 64), 1u, __ATOMIC_RELAXED, __HIP_MEMORY_SCOPE_AGENT) + 1u;
    __syncthreads();
    const unsigned call = __builtin_amdgcn_readfirstlane(*cw);
    if (blockIdx.x < NSEND) comm(P, blockIdx.x, call, lds);
    else gemm(P, blockIdx.x - NSEND, call, lds);
}

}

extern "C" void kernel_launch(int rank, int nranks, ncclComm_t comm,
                              hipStream_t stream,
                              void* const* d_in, const int* in_sizes, int n_in,
                              void* d_out, int out_size,
                              const int* in_dtypes, int out_dtype,
                              void* const* d_out_all,
                              void* const* d_ws_all, size_t ws_bytes) {
    (void)comm; (void)in_dtypes; (void)out_dtype; (void)d_out;
    if (nranks != NR || n_in != 2 || rank < 0 || rank >= NR || in_sizes[0] != MM * KL || in_sizes[1] != KL * NN ||
        out_size != ML * NN || ws_bytes < WS_USED) {
        fprintf(stderr, "kernel_launch: unexpected problem shape\n");
        return;
    }
    Params P;
    P.x = (const unsigned short*)d_in[0];
    P.w = (const unsigned short*)d_in[1];
    for (int p = 0; p < NR; ++p) { P.ws[p] = (unsigned char*)d_ws_all[p]; P.out[p] = (signed char*)d_out_all[p]; }
    P.rank = rank; P.pad = 0;
    HIP_CHECK(hipFuncSetAttribute((const void*)gemm_rs_q8_kernel, hipFuncAttributeMaxDynamicSharedMemorySize, LDS_BYTES));
    hipLaunchKernelGGL(gemm_rs_q8_kernel, dim3(NBLK), dim3(NTHR), LDS_BYTES, stream, P);
    HIP_CHECK(hipGetLastError());
}
```

```cpp
#include <hip/hip_runtime.h>
#include <rccl/rccl.h>
#include <cstdio>
#include <cstdint>

#define HIP_CHECK(expr) do { hipError_t _e = (expr); if (_e != hipSuccess) { \
    fprintf(stderr, "HIP error %s at %s:%d\n", hipGetErrorString(_e), __FILE__, __LINE__); \
    return; } } while (0)

namespace {

#define LAS __attribute__((address_space(3)))
#define GAS __attribute__((address_space(1)))
typedef short bf16x8 __attribute__((ext_vector_type(8)));
typedef short s16x4 __attribute__((ext_vector_type(4)));
typedef float f32x4 __attribute__((ext_vector_type(4)));
typedef unsigned u32x4 __attribute__((ext_vector_type(4)));
typedef unsigned u32x2 __attribute__((ext_vector_type(2)));
typedef GAS unsigned gu32;
typedef GAS unsigned long long gu64;

constexpr int NR = 8;
constexpr int MM = 4096, NN = 16384;
constexpr int KL = 512;
constexpr int NL = NN / NR;
constexpr int ML = MM / NR;
constexpr int NSL = KL / 64;
constexpr int NRB = MM / 256;
constexpr int NCB = NL / 256;
constexpr int NKT = NR * NSL;
constexpr int CHUNK = 32768;
constexpr int NSEND = 128, NGEMM = 128, NBLK = NSEND + NGEMM, NTHR = 512;

constexpr int NPC = NSL * 24 * 4;
constexpr int PK_E = 4096, PK_E2 = 5120, PK_NAR = 5632, PK_WID = 6144, PK_GR = 6400, SLOT = 6528;
constexpr int NAR_CAP = 256, WID_CAP = 64;
constexpr size_t OFF_A = 0;
constexpr size_t OFF_B = (size_t)NR * NSL * NRB * CHUNK;
constexpr size_t OFF_P = OFF_B + (size_t)NR * NSL * NCB * CHUNK;
constexpr size_t OFF_C = OFF_P + (size_t)NR * NPC * SLOT;
constexpr size_t C_FA = OFF_C;
constexpr size_t C_FB = C_FA + NRB * NR * 128;
constexpr size_t C_AMAX = C_FB + NCB * NR * 128;
constexpr size_t C_DONE = C_AMAX + NR * NGEMM * 8;
constexpr size_t C_CNT = C_DONE + NR * 128;
constexpr size_t C_TICK = C_CNT + NBLK * 8 * 16;
constexpr size_t C_TMO = C_TICK + 2 * NR * 128;
constexpr size_t C_STAMP = C_TMO + 128;
constexpr int NSTAMP = 128;
constexpr size_t WS_USED = C_STAMP + NBLK * NSTAMP * 8;
static_assert(OFF_C % 128 == 0 && SLOT % 128 == 0, "alignment");

constexpr int LDS_STAGE = 131072;
constexpr int LDS_CTL = LDS_STAGE;
constexpr int LDS_BYTES = LDS_STAGE + 1024;
constexpr unsigned long long DEADLINE = 40000000ull;

struct Params {
    const unsigned short* x;
    const unsigned short* w;
    unsigned char* ws[NR];
    signed char* out[NR];
    int rank;
    int pad;
};

__device__ __forceinline__ unsigned long long now() { return __builtin_amdgcn_s_memrealtime(); }
__device__ __forceinline__ unsigned ld_sys32(const void* p) { return __hip_atomic_load((gu32*)p, __ATOMIC_RELAXED, __HIP_MEMORY_SCOPE_SYSTEM); }
__device__ __forceinline__ void st_sys32(void* p, unsigned v) { __hip_atomic_store((gu32*)p, v, __ATOMIC_RELAXED, __HIP_MEMORY_SCOPE_SYSTEM); }
__device__ __forceinline__ unsigned ld_agent32(const void* p) { return __hip_atomic_load((gu32*)p, __ATOMIC_RELAXED, __HIP_MEMORY_SCOPE_AGENT); }
__device__ __forceinline__ void st_agent32(void* p, unsigned v) { __hip_atomic_store((gu32*)p, v, __ATOMIC_RELAXED, __HIP_MEMORY_SCOPE_AGENT); }
__device__ __forceinline__ void st_wt16(void* p, u32x4 v) {
    asm volatile("global_store_dwordx4 %0, %1, off sc0 sc1\n\ts_nop 1" :: "v"(p), "v"(v) : "memory");
}
template <class T> __device__ __forceinline__ T* uni(T* p) {
    const unsigned long long v = (unsigned long long)p;
    const unsigned lo = __builtin_amdgcn_readfirstlane((unsigned)v), hi = __builtin_amdgcn_readfirstlane((unsigned)(v >> 32));
    return (T*)(((unsigned long long)hi << 32) | lo);
}
#define WAIT_V(n) asm volatile("s_waitcnt vmcnt(" #n ")" ::: "memory")
#define WAIT_L(n) asm volatile("s_waitcnt lgkmcnt(" #n ")" ::: "memory")
#define BAR __builtin_amdgcn_s_barrier()
#define SCHED __builtin_amdgcn_sched_barrier(0)
#define CFENCE asm volatile("" ::: "memory")

#define GAUX 17
#define STAMP(ws_own, idx) do {} while (0)
#define WSTAMP(ws_own, idx) do {} while (0)

__device__ __forceinline__ unsigned wave_umax(unsigned v) {
    v = max(v, (unsigned)__builtin_amdgcn_update_dpp(0, (int)v, 0x111, 0xf, 0xf, false));
    v = max(v, (unsigned)__builtin_amdgcn_update_dpp(0, (int)v, 0x112, 0xf, 0xf, false));
    v = max(v, (unsigned)__builtin_amdgcn_update_dpp(0, (int)v, 0x114, 0xf, 0xf, false));
    v = max(v, (unsigned)__builtin_amdgcn_update_dpp(0, (int)v, 0x118, 0xf, 0xf, false));
    v = max(v, (unsigned)__builtin_amdgcn_update_dpp(0, (int)v, 0x142, 0xa, 0xf, false));
    v = max(v, (unsigned)__builtin_amdgcn_update_dpp(0, (int)v, 0x143, 0xc, 0xf, false));
    return (unsigned)__builtin_amdgcn_readlane((int)v, 63);
}

#define ROT_SHIFT 0
__device__ __forceinline__ int rot_of(int d) { return d == 0 ? 0 : (((d - 1) * 16) / 7 + ROT_SHIFT) & 15; }
__device__ __forceinline__ int link_of(int xcd) { return xcd; }

__device__ __forceinline__ bool give_up(unsigned long long t0, unsigned char* ws_own, unsigned call) {
    if (ld_agent32(ws_own + C_TMO) == call || now() - t0 > DEADLINE) { st_agent32(ws_own + C_TMO, call); return true; }
    return false;
}

#define SEND_CUS 6
#define RAW_TAIL 0
constexpr int WAVE_LDS = 64 * 144 + NAR_CAP * 2 + WID_CAP * 4;
typedef unsigned short u16x2 __attribute__((ext_vector_type(2)));
__device__ __forceinline__ u16x2 pk(unsigned v) { return __builtin_bit_cast(u16x2, v); }
__device__ __forceinline__ unsigned up(u16x2 v) { return __builtin_bit_cast(unsigned, v); }
template <int OFF> __device__ __forceinline__ void st_wt16o(void* p, u32x4 v) {
    asm volatile("global_store_dwordx4 %0, %1, off offset:%2 sc0 sc1\n\ts_nop 1" :: "v"(p), "v"(v), "i"(OFF) : "memory");
}
__device__ __forceinline__ void add_sys32(void* p) { (void)__hip_atomic_fetch_add((gu32*)p, 1u, __ATOMIC_RELAXED, __HIP_MEMORY_SCOPE_SYSTEM); }

struct PieceId { int isB, s, idx, pc; };
__device__ __forceinline__ PieceId piece_of(int q, int rot) {
    const int c = q >> 2, s = c / 24, u = c - 24 * s;
    PieceId id; id.isB = u < NCB; id.s = s; id.idx = id.isB ? u : ((u - NCB + rot) & 15); id.pc = q & 3; return id;
}
__device__ __forceinline__ unsigned char* image_of(unsigned char* ws, int src, const PieceId& id, int lane, int& stride4) {
    stride4 = id.isB ? 16384 : 4096;
    return id.isB ? ws + OFF_B + (size_t)((src * NSL + id.s) * NCB + id.idx) * CHUNK + id.pc * 4096 + lane * 16
                  : ws + OFF_A + (size_t)((src * NSL + id.s) * NRB + id.idx) * CHUNK + id.pc * 8192 + lane * 16;
}
__device__ __forceinline__ unsigned char* counter_of(unsigned char* ws, int src, const PieceId& id) {
    return (id.isB ? ws + C_FB : ws + C_FA) + (size_t)(id.idx * NR + src) * 128 + id.s * 4;
}
__device__ __forceinline__ void store_image(unsigned char* img, int stride4, const u32x4 (&v)[8]) {
    st_wt16o<0>(img, v[0]); st_wt16o<1024>(img, v[1]); st_wt16o<2048>(img, v[2]); st_wt16o<3072>(img, v[3]);
    unsigned char* i2 = img + stride4;
    st_wt16o<0>(i2, v[4]); st_wt16o<1024>(i2, v[5]); st_wt16o<2048>(i2, v[6]); st_wt16o<3072>(i2, v[7]);
}
__device__ __forceinline__ void issue_piece(const Params& P, int dest, const PieceId& id, int lane, u32x4 (&r)[8]) {
    if (id.isB) {
        const unsigned char* src = (const unsigned char*)P.w + (size_t)(64 * id.s + (lane >> 3)) * (NN * 2) + (size_t)(NL * dest + 256 * id.idx + 64 * id.pc + (lane & 7) * 8) * 2;
#pragma unroll
        for (int it = 0; it < 8; ++it) r[it] = *(const GAS u32x4*)(src + (size_t)(it * 8) * (NN * 2));
    } else {
        const unsigned char* src = (const unsigned char*)P.x + (size_t)(256 * id.idx + 64 * id.pc + (lane & 15)) * (KL * 2) + (64 * id.s + 8 * (lane >> 4)) * 2;
#pragma unroll
        for (int ff = 0; ff < 8; ++ff) r[ff] = *(const GAS u32x4*)(src + (size_t)(16 * (ff >> 1)) * (KL * 2) + (ff & 1) * 64);
    }
}
__device__ __forceinline__ void finish_piece(const PieceId& id, int lane, LAS unsigned char* wl, u32x4 (&v)[8]) {
    if (id.isB) {
#pragma unroll
        for (int it = 0; it < 8; ++it) *(LAS u32x4*)(wl + (it * 8 + (lane >> 3)) * 144 + (lane & 7) * 16) = v[it];
        WAIT_L(0);
        const int fg = lane >> 4, q4 = (lane >> 2) & 3, p4 = lane & 3;
        LAS unsigned char* a0 = wl + (8 * fg + q4) * 144 + p4 * 32;
#pragma unroll
        for (int ff = 0; ff < 8; ++ff) {
            const int hn = ff >> 1, kf = ff & 1;
            LAS unsigned char* a = a0 + (32 * kf) * 144 + hn * 8;
            typedef short v4i16_t __attribute__((ext_vector_type(4)));
            const v4i16_t lo = __builtin_amdgcn_ds_read_tr16_b64_v4i16((LAS v4i16_t*)a);
            const v4i16_t hi = __builtin_amdgcn_ds_read_tr16_b64_v4i16((LAS v4i16_t*)(a + 4 * 144));
            const u32x2 l2 = __builtin_bit_cast(u32x2, lo), h2 = __builtin_bit_cast(u32x2, hi);
            v[ff] = (u32x4){l2.x, l2.y, h2.x, h2.y};
        }
        WAIT_L(0);
    }
}

constexpr unsigned META_RAW = 0x100u;
__device__ __forceinline__ bool pack_piece(const u32x4 (&v)[8], int lane, LAS unsigned char* wl, unsigned char* sl, unsigned& meta) {
    u16x2 mx = pk(0u);
#pragma unroll
    for (int ff = 0; ff < 8; ++ff) {
        *(LAS u32x4*)(wl + ff * 1024 + lane * 16) = v[ff];
#pragma unroll
        for (int i = 0; i < 4; ++i) mx = __builtin_elementwise_max(mx, pk(v[ff][i] & 0x7fff7fffu));
    }
    const unsigned hi = wave_umax(max((unsigned)mx.x, (unsigned)mx.y)) >> 7;
    const unsigned base = hi > 7u ? hi - 7u : 0u, base2 = base * 0x00010001u;
    u32x4 sm[4]; unsigned c12[8], mk[2] = {0u, 0u};
#pragma unroll
    for (int ff = 0; ff < 8; ++ff) {
        unsigned n = 0, t[4];
#pragma unroll
        for (int i = 0; i < 4; ++i) {
            const unsigned dw = v[ff][i];
            const u16x2 ex = pk((dw >> 7) & 0x00ff00ffu);
            n |= up(__builtin_elementwise_sub_sat(ex, pk(base2))) << (3 * i);
            mk[ff >> 2] |= up(__builtin_elementwise_min(__builtin_elementwise_sub_sat(pk(base2), ex), pk(0x00010001u))) << (4 * (ff & 3) + i);
            t[i] = (dw & 0x007f007fu) | ((dw >> 8) & ~0x007f007fu);
        }
        sm[ff >> 1][(ff & 1) * 2] = __builtin_amdgcn_perm(t[1], t[0], 0x06040200u);
        sm[ff >> 1][(ff & 1) * 2 + 1] = __builtin_amdgcn_perm(t[3], t[2], 0x06040200u);
        c12[ff] = n;
    }
    LAS unsigned short* nar = (LAS unsigned short*)(wl + 8192 + 512);
    LAS unsigned* wid = (LAS unsigned*)(wl + 9216);
    unsigned nn = 0, nw = 0;
    while (__ballot((mk[0] | mk[1]) != 0u)) {
        const bool act = (mk[0] | mk[1]) != 0u, g = mk[0] == 0u;
        const unsigned cur = g ? mk[1] : mk[0];
        const unsigned bit = act ? (unsigned)__builtin_ctz(cur) : 0u;
        const unsigned idx = (((g ? 4u : 0u) + ((bit >> 2) & 3u)) * 64u + (unsigned)lane) * 8u + 2u * (bit & 3u) + (bit >> 4);
        const unsigned val = *(LAS unsigned short*)(wl + idx * 2u);
        const unsigned d = base - ((val >> 7) & 0xffu);
        const bool wide = act && d > 15u;
        const unsigned long long bn = __ballot(act && !wide), bw = __ballot(wide);
        const unsigned pos = nn + __builtin_amdgcn_mbcnt_hi((unsigned)(bn >> 32), __builtin_amdgcn_mbcnt_lo((unsigned)bn, 0u));
        if (act && !wide && pos < (unsigned)NAR_CAP) nar[pos] = (unsigned short)((idx << 4) | d);
        nn += (unsigned)__builtin_popcountll(bn);
        if (bw) {
            const unsigned pw = nw + __builtin_amdgcn_mbcnt_hi((unsigned)(bw >> 32), __builtin_amdgcn_mbcnt_lo((unsigned)bw, 0u));
            if (wide && pw < (unsigned)WID_CAP) wid[pw] = (idx << 16) | val;
            nw += (unsigned)__builtin_popcountll(bw);
        }
        if (g) mk[1] = cur & (cur - 1u); else mk[0] = cur & (cur - 1u);
        if (nn > (unsigned)NAR_CAP || nw > (unsigned)WID_CAP) return false;
    }
    unsigned char* a = sl + lane * 16;
    st_wt16o<0>(a, sm[0]); st_wt16o<1024>(a, sm[1]); st_wt16o<2048>(a, sm[2]); st_wt16o<3072>(a, sm[3]);
    unsigned w[6];
#pragma unroll
    for (int g = 0; g < 2; ++g) {
        const u16x2 ca = pk(c12[4 * g]), cb = pk(c12[4 * g + 1]), cc = pk(c12[4 * g + 2]), cd = pk(c12[4 * g + 3]);
        w[3 * g] = up(ca | (cb << (unsigned short)12));
        w[3 * g + 1] = up((cb >> (unsigned short)4) | (cc << (unsigned short)8));
        w[3 * g + 2] = up((cc >> (unsigned short)8) | (cd << (unsigned short)4));
    }
    st_wt16o<0>(a + PK_E, (u32x4){w[0], w[1], w[2], w[3]});
    *(LAS u32x2*)(wl + 8192 + lane * 8) = (u32x2){w[4], w[5]};
    WAIT_L(0);
    if (16u * lane < 512u + 2u * nn) st_wt16o<0>(a + PK_E2, *(volatile LAS u32x4*)(wl + 8192 + lane * 16));
    if (4u * lane < nw) st_wt16o<0>(a + PK_WID, *(volatile LAS u32x4*)(wl + 9216 + lane * 16));
    meta = base | (nn << 9) | (nw << 18);
    return true;
}
__device__ __forceinline__ void unpack_piece(const unsigned char* sl, unsigned meta, int lane, LAS unsigned char* wl, u32x4 (&v)[8]) {
    u32x4 k0, k1, k2, k3, e4; u32x2 e2; unsigned na, nb, wd;
    const unsigned char* a = sl + lane * 16;
    asm volatile("global_load_dwordx4 %0, %9, off sc0 sc1\n\t"
                 "global_load_dwordx4 %1, %9, off offset:1024 sc0 sc1\n\t"
                 "global_load_dwordx4 %2, %9, off offset:2048 sc0 sc1\n\t"
                 "global_load_dwordx4 %3, %9, off offset:3072 sc0 sc1\n\t"
                 "global_load_dwordx4 %4, %10, off sc0 sc1\n\t"
                 "global_load_dwordx2 %5, %11, off sc0 sc1\n\t"
                 "global_load_dword %6, %12, off sc0 sc1\n\t"
                 "global_load_dword %7, %12, off offset:256 sc0 sc1\n\t"
                 "global_load_dword %8, %12, off offset:512 sc0 sc1\n\t"
                 "s_waitcnt vmcnt(0)"
                 : "=&v"(k0), "=&v"(k1), "=&v"(k2), "=&v"(k3), "=&v"(e4), "=&v"(e2), "=&v"(na), "=&v"(nb), "=&v"(wd)
                 : "v"(a), "v"(a + PK_E), "v"(sl + PK_E2 + lane * 8), "v"(sl + PK_NAR + lane * 4) : "memory");
    static_assert(PK_WID == PK_NAR + 512, "list offsets");
    const u32x4 sm[4] = {k0, k1, k2, k3};
    const unsigned w[6] = {e4.x, e4.y, e4.z, e4.w, e2.x, e2.y};
    unsigned c12[8];
#pragma unroll
    for (int g = 0; g < 2; ++g) {
        const u16x2 w0 = pk(w[3 * g]), w1 = pk(w[3 * g + 1]), w2 = pk(w[3 * g + 2]);
        c12[4 * g] = up(w0);
        c12[4 * g + 1] = up((w0 >> (unsigned short)12) | (w1 << (unsigned short)4));
        c12[4 * g + 2] = up((w1 >> (unsigned short)8) | (w2 << (unsigned short)8));
        c12[4 * g + 3] = up(w2 >> (unsigned short)4);
    }
    const unsigned base7 = ((meta & 0xffu) << 7) * 0x00010001u;
#pragma unroll
    for (int ff = 0; ff < 8; ++ff) {
        u32x4 o;
#pragma unroll
        for (int i = 0; i < 4; ++i) {
            const unsigned s4 = sm[ff >> 1][(ff & 1) * 2 + (i >> 1)];
            const unsigned u = __builtin_amdgcn_perm(s4, s4, (i & 1) ? 0x03030202u : 0x01010000u);
            const unsigned e = (((c12[ff] >> (3 * i)) & 0x00070007u) << 7) + base7;
            o[i] = (u & 0x807f807fu) | e;
        }
        *(LAS u32x4*)(wl + ff * 1024 + lane * 16) = o;
    }
    const unsigned nn = (meta >> 9) & 0x1ffu, nw = meta >> 18;
#pragma unroll
    for (int r = 0; r < 4; ++r) {
        const unsigned e = (unsigned)((r >> 1) * 128 + 2 * lane + (r & 1));
        if (e < nn) {
            const unsigned ent = ((r >> 1 ? nb : na) >> (16 * (r & 1))) & 0xffffu, idx = ent >> 4;
            (void)__hip_atomic_fetch_sub((LAS unsigned*)(wl + (idx >> 1) * 4), ((ent & 15u) << 7) << (16 * (idx & 1u)), __ATOMIC_RELAXED, __HIP_MEMORY_SCOPE_WORKGROUP);
        }
    }
    if ((unsigned)lane < nw) *(LAS unsigned short*)(wl + ((wd >> 16) & 4095u) * 2) = (unsigned short)wd;
    WAIT_L(0);
#pragma unroll
    for (int ff = 0; ff < 8; ++ff) v[ff] = *(LAS u32x4*)(wl + ff * 1024 + lane * 16);
}

constexpr int NUCU = 16 - SEND_CUS;
constexpr int NUW = NUCU * 8 + 18;
constexpr int NSELF = 128 + 7 * NUCU * 8;

__device__ __forceinline__ void comm(const Params& P, int b, unsigned callv, LAS unsigned char* lds) {
    const int tid = threadIdx.x, wid = __builtin_amdgcn_readfirstlane(tid >> 6), lane = tid & 63;
    const int rank = P.rank;
    const int xcd = b & 7, slot = b >> 3;
    unsigned char* own = P.ws[rank];
    LAS unsigned char* wl = lds + wid * WAVE_LDS;
    STAMP(own, 0);
    if (xcd != 0 && slot < SEND_CUS) {
        const int d = link_of(xcd), dest = (rank + d) & 7, rot = rot_of(d);
        constexpr int nw = SEND_CUS * 8;
        unsigned char* wsd = P.ws[dest];
        gu32* tick = (gu32*)(own + C_TICK + (size_t)d * 128);
        u32x4 nxt[8];
        int q = slot * 8 + wid;
        issue_piece(P, dest, piece_of(q, rot), lane, nxt);
        const unsigned call = __builtin_amdgcn_readfirstlane(callv);
        const unsigned tbase = (call - 1u) * (unsigned)NPC - (unsigned)nw;
        unsigned tk = 0;
        if (lane == 0) tk = __hip_atomic_fetch_add(tick, 1u, __ATOMIC_RELAXED, __HIP_MEMORY_SCOPE_AGENT);
        unsigned char* pub = nullptr; unsigned long long pubv = 0; bool pubdirect = false;
        for (;;) {
            const PieceId id = piece_of(q, rot);
            u32x4 v[8];
            asm volatile("s_waitcnt vmcnt(0)" : "+v"(nxt[0]), "+v"(nxt[1]), "+v"(nxt[2]), "+v"(nxt[3]), "+v"(nxt[4]), "+v"(nxt[5]), "+v"(nxt[6]), "+v"(nxt[7]), "+v"(tk) :: "memory");
            if (pub && lane == 0) {
                if (pubdirect) add_sys32(pub);
                else __hip_atomic_store((gu64*)pub, pubv, __ATOMIC_RELAXED, __HIP_MEMORY_SCOPE_SYSTEM);
            }
#pragma unroll
            for (int ff = 0; ff < 8; ++ff) v[ff] = nxt[ff];
            const unsigned qn = (unsigned)__builtin_amdgcn_readfirstlane(tk) - tbase;
            const bool more = qn < (unsigned)NPC;
            if (more) {
                issue_piece(P, dest, piece_of((int)qn, rot), lane, nxt);
                if (lane == 0) tk = __hip_atomic_fetch_add(tick, 1u, __ATOMIC_RELAXED, __HIP_MEMORY_SCOPE_AGENT);
            }
            finish_piece(id, lane, wl, v);
            const bool direct = q >= NPC - RAW_TAIL;
            unsigned meta = META_RAW;
            unsigned char* sl = wsd + OFF_P + (size_t)(rank * NPC + q) * SLOT;
            if (direct || !pack_piece(v, lane, wl, sl, meta)) { meta = META_RAW; int st4; unsigned char* img = image_of(wsd, rank, id, lane, st4); store_image(img, st4, v); }
            pubdirect = direct;
            pub = direct ? counter_of(wsd, rank, id) : sl + PK_GR;
            pubv = ((unsigned long long)call << 32) | meta;
            if (!more) break;
            q = (int)qn;
        }
        WAIT_V(0);
        if (pub && lane == 0) {
            if (pubdirect) add_sys32(pub);
            else __hip_atomic_store((gu64*)pub, pubv, __ATOMIC_RELAXED, __HIP_MEMORY_SCOPE_SYSTEM);
        }
        STAMP(own, 1); WSTAMP(own, 8);
    } else {
        const int g0 = xcd == 0 ? slot * 8 + wid : 128 + ((xcd - 1) * NUCU + slot - SEND_CUS) * 8 + wid;
        for (int q = g0; q < NPC; q += NSELF) {
            const PieceId id = piece_of(q, 0);
            u32x4 v[8];
            issue_piece(P, rank, id, lane, v);
            finish_piece(id, lane, wl, v);
            int st4; unsigned char* img = image_of(own, rank, id, lane, st4);
            store_image(img, st4, v);
            WAIT_V(0);
            if (lane == 0) add_sys32(counter_of(own, rank, id));
        }
        STAMP(own, 11);
        const unsigned call = __builtin_amdgcn_readfirstlane(callv);
        const int d = xcd != 0 ? link_of(xcd) : 1 + g0 % 7;
        const int first = xcd != 0 ? (slot - SEND_CUS) * 8 + wid : NUCU * 8 + g0 / 7;
        const int src = (rank - d) & 7, rot = rot_of(d);
        constexpr int NUP = NPC - RAW_TAIL;
        gu32* tick = (gu32*)(own + C_TICK + (size_t)(NR + d) * 128);
        const unsigned tbase = (call - 1u) * (unsigned)NUP - (unsigned)NUW;
        unsigned tk = 0;
        for (int q = first < NUW ? first : NUP; q < NUP;) {
            if (lane == 0) tk = __hip_atomic_fetch_add(tick, 1u, __ATOMIC_RELAXED, __HIP_MEMORY_SCOPE_AGENT);
            const PieceId id = piece_of(q, rot);
            const unsigned char* sl = own + OFF_P + (size_t)(src * NPC + q) * SLOT;
            unsigned meta;
            {
                const unsigned long long t0 = now();
                for (;;) {
                    const unsigned long long g = __hip_atomic_load((gu64*)(sl + PK_GR), __ATOMIC_RELAXED, __HIP_MEMORY_SCOPE_SYSTEM);
                    const unsigned tag = __builtin_amdgcn_readfirstlane((unsigned)(g >> 32));
                    meta = __builtin_amdgcn_readfirstlane((unsigned)g);
                    if (tag >= call) break;
                    if (give_up(t0, own, call)) { meta = META_RAW; break; }
                    __builtin_amdgcn_s_sleep(2);
                }
            }
            if (!(meta & META_RAW)) {
                u32x4 v[8];
                unpack_piece(sl, meta, lane, wl, v);
                int st4; unsigned char* img = image_of(own, src, id, lane, st4);
                store_image(img, st4, v);
                WAIT_V(0);
            }
            if (lane == 0) add_sys32(counter_of(own, src, id));
            const unsigned qn = (unsigned)__builtin_amdgcn_readfirstlane(tk) - tbase;
            q = qn < (unsigned)NUP ? (int)qn : NUP;
        }
        STAMP(own, 1); WSTAMP(own, 8);
    }
    if (b == 0) {
        const unsigned call = __builtin_amdgcn_readfirstlane(callv);
        if (wid < 2) {
            const unsigned char* f = own + C_DONE + (size_t)(tid >> 4) * 128 + (tid & 15) * 4;
            const unsigned long long t0 = now();
            bool bad = false;
            while (!__all(ld_sys32(f) >= call)) {
                if (give_up(t0, own, call)) { bad = true; break; }
                __builtin_amdgcn_s_sleep(2);
            }
            if (bad && lane == 0) P.out[rank][rank * NL] = (signed char)-128;
        }
        STAMP(own, 2);
    }
}

__device__ __forceinline__ void gemm(const Params& P, int b, unsigned call, LAS unsigned char* lds) {
    const int tid = threadIdx.x, wid = __builtin_amdgcn_readfirstlane(tid >> 6), lane = tid & 63, wr = wid >> 2, wc = wid & 3;
    const int rank = P.rank;
    const int xcd = b & 7, slot = b >> 3;
    const int ti = 2 * ((rank + xcd) & 7) + (slot >> 3), tj = slot & 7;
    unsigned char* own = P.ws[rank];
    STAMP(own, 0);
    const unsigned char* abase = uni(own + OFF_A + (size_t)ti * CHUNK);
    const unsigned char* bbase = uni(own + OFF_B + (size_t)tj * CHUNK);
    auto aptr = [&](int k) -> const unsigned char* { return abase + ((size_t)(unsigned)k << 19); };
    auto bptr = [&](int k) -> const unsigned char* { return bbase + ((size_t)(unsigned)k << 18); };
    static_assert(NRB * CHUNK == (1 << 19) && NCB * CHUNK == (1 << 18), "chunk strides");

    LAS unsigned* ctl = (LAS unsigned*)(lds + LDS_CTL);
    const unsigned ctl_addr = (unsigned)(size_t)ctl;
    const unsigned char* fbaseA = uni(own + C_FA + (size_t)ti * NR * 128);
    const unsigned char* fbaseB = uni(own + C_FB + (size_t)tj * NR * 128);
    const unsigned foff = (unsigned)(lane >> 3) * 128u + (unsigned)(lane & 7) * 4u;
    const unsigned target = 4u * call;
    unsigned long long known = 0, taken = 0;
    auto pick = [&]() -> int {
        if ((known & ~taken) == 0) {
            if (taken == ~0ull) return 0;
            if (wid == 0) {
                const unsigned long long t0 = now();
                unsigned long long m;
                for (;;) {
                    unsigned va, vb;
                    asm volatile("global_load_dword %0, %2, %3 sc0 sc1\n\tglobal_load_dword %1, %2, %4 sc0 sc1\n\ts_waitcnt vmcnt(0)"
                                 : "=&v"(va), "=&v"(vb) : "v"(foff), "s"(fbaseA), "s"(fbaseB) : "memory");
                    m = __ballot(va >= target && vb >= target);
                    if (m & ~taken) break;
                    if (give_up(t0, own, call)) { m = ~0ull; break; }
                    __builtin_amdgcn_s_sleep(1);
                }
                const u32x2 mv = {(unsigned)m, (unsigned)(m >> 32)};
                asm volatile("ds_write_b64 %0, %1\n\ts_waitcnt lgkmcnt(0)" :: "v"(ctl_addr), "v"(mv) : "memory");
            }
            CFENCE; BAR; CFENCE;
            u32x2 r;
            asm volatile("ds_read_b64 %0, %1\n\ts_waitcnt lgkmcnt(0)" : "=v"(r) : "v"(ctl_addr) : "memory");
            known |= ((unsigned long long)(unsigned)__builtin_amdgcn_readfirstlane(r.y) << 32) | (unsigned)__builtin_amdgcn_readfirstlane(r.x);
        }
        const int k = __builtin_ctzll(known & ~taken);
        taken |= 1ull << k;
        return k;
    };
    unsigned char* amax_dst = P.ws[lane & 7] + C_AMAX + (size_t)(rank * NGEMM + b) * 8;
    asm volatile("" : "+v"(amax_dst));
    const int dest = ti >> 1;
    signed char* out_tile = uni(P.out[dest] + (size_t)((ti & 1) * 256) * NN + (size_t)rank * NL + tj * 256);
    unsigned char* done_flag = uni(P.ws[dest] + C_DONE + (size_t)rank * 128 + ((ti & 1) * 8 + tj) * 4);

    f32x4 acc[2][2][4][2];
#pragma unroll
    for (int a = 0; a < 2; ++a)
#pragma unroll
        for (int c = 0; c < 2; ++c)
#pragma unroll
            for (int m = 0; m < 4; ++m)
#pragma unroll
                for (int n = 0; n < 2; ++n) acc[a][c][m][n] = (f32x4){0.f, 0.f, 0.f, 0.f};
    bf16x8 At[4][2], B0[2][2], B1[2][2];
    unsigned voff[2] = {(unsigned)tid * 16u, (unsigned)tid * 16u + 8192u};
    asm volatile("" : "+v"(voff[0]), "+v"(voff[1]));
    const unsigned ldsw = (unsigned)wid * 1024u;
    const int aoff = wr * 8192 + lane * 16, boff = wc * 4096 + lane * 16;
    constexpr int HTB = 16384;
#define SA(b_, h_) (((b_) * 2 + (h_)) * HTB)
#define SB(b_, h_) ((4 + (b_) * 2 + (h_)) * HTB)
#define STAGE(bufoff, gbase) do { _Pragma("unroll") for (int _i = 0; _i < 2; ++_i) \
        __builtin_amdgcn_global_load_lds((const unsigned*)((const char*)(gbase) + voff[_i]), (LAS unsigned*)(lds + (bufoff) + ldsw + _i * 8192), 16, 0, GAUX); } while (0)
#define LDA(dst, b_, h_) do { _Pragma("unroll") for (int m = 0; m < 4; ++m) _Pragma("unroll") for (int k = 0; k < 2; ++k) dst[m][k] = *(const LAS bf16x8*)(lds + SA(b_, h_) + aoff + m * 2048 + k * 1024); } while (0)
#define LDB(dst, b_, h_) do { _Pragma("unroll") for (int n = 0; n < 2; ++n) _Pragma("unroll") for (int k = 0; k < 2; ++k) dst[n][k] = *(const LAS bf16x8*)(lds + SB(b_, h_) + boff + n * 2048 + k * 1024); } while (0)
#define MMA(ai, bj, At_, Bt_) do { __builtin_amdgcn_s_setprio(1); _Pragma("unroll") for (int m = 0; m < 4; ++m) _Pragma("unroll") for (int n = 0; n < 2; ++n) _Pragma("unroll") for (int k = 0; k < 2; ++k) \
        acc[ai][bj][m][n] = __builtin_amdgcn_mfma_f32_16x16x32_bf16(Bt_[n][k], At_[m][k], acc[ai][bj][m][n], 0, 0, 0); __builtin_amdgcn_s_setprio(0); } while (0)

    {
        const int k0 = pick();
        const unsigned char* a0 = aptr(k0); const unsigned char* b0 = bptr(k0);
        STAGE(SB(0, 0), b0); STAGE(SA(0, 0), a0); STAGE(SB(0, 1), b0 + HTB); STAGE(SA(0, 1), a0 + HTB);
    }
    int k1 = pick();
    if (wr == 1) BAR;
    WAIT_V(4); BAR;
    {
        const unsigned char* a1 = aptr(k1); const unsigned char* b1 = bptr(k1);
        STAGE(SB(1, 0), b1); STAGE(SA(1, 0), a1); STAGE(SB(1, 1), b1 + HTB);
    }
    WAIT_V(6); BAR;
    STAMP(own, 1);
    for (int t = 0; t < NKT; t += 2) {
        const unsigned char* a1 = aptr(k1);
        unsigned pa = 0, pb = 0;
        const bool look = wid == 0 && known != ~0ull;
        if (look) asm volatile("global_load_dword %0, %2, %3 sc0 sc1\n\tglobal_load_dword %1, %2, %4 sc0 sc1" : "=&v"(pa), "=&v"(pb) : "v"(foff), "s"(fbaseA), "s"(fbaseB) : "memory");
        LDB(B0, 0, 0); SCHED; LDA(At, 0, 0); STAGE(SA(1, 1), a1 + HTB);
        WAIT_L(8); BAR; WAIT_L(0); MMA(0, 0, At, B0); BAR; SCHED;
        const int k2 = pick();
        const unsigned char* a2 = aptr(k2); const unsigned char* b2 = bptr(k2);
        LDB(B1, 0, 1); STAGE(SB(0, 0), b2);
        BAR; WAIT_L(0); MMA(0, 1, At, B1); BAR;
        LDA(At, 0, 1); STAGE(SA(0, 0), a2);
        BAR; WAIT_L(0); MMA(1, 0, At, B0); BAR; SCHED;
        STAGE(SB(0, 1), b2 + HTB);
        WAIT_V(6);
        if (wid == 0) {
            unsigned long long m = known;
            if (look) { asm volatile("" : "+v"(pa), "+v"(pb)); m |= __ballot(pa >= target && pb >= target); }
            const u32x2 mv = {(unsigned)m, (unsigned)(m >> 32)};
            asm volatile("ds_write_b64 %0, %1 offset:8\n\ts_waitcnt lgkmcnt(0)" :: "v"(ctl_addr), "v"(mv) : "memory");
        }
        BAR; MMA(1, 1, At, B1); BAR;
        u32x2 rn;
        asm volatile("ds_read_b64 %0, %1 offset:8" : "=v"(rn) : "v"(ctl_addr) : "memory");
        LDB(B0, 1, 0); SCHED; LDA(At, 1, 0); STAGE(SA(0, 1), a2 + HTB);
        WAIT_L(8); BAR; WAIT_L(0);
        asm volatile("" : "+v"(rn));
        known |= ((unsigned long long)(unsigned)__builtin_amdgcn_readfirstlane(rn.y) << 32) | (unsigned)__builtin_amdgcn_readfirstlane(rn.x);
        MMA(0, 0, At, B0); BAR; SCHED;
        const int k3 = pick();
        const unsigned char* a3 = aptr(k3); const unsigned char* b3 = bptr(k3);
        LDB(B1, 1, 1); STAGE(SB(1, 0), b3);
        BAR; WAIT_L(0); MMA(0, 1, At, B1); BAR;
        LDA(At, 1, 1); STAGE(SA(1, 0), a3);
        BAR; WAIT_L(0); MMA(1, 0, At, B0); BAR; SCHED;
        STAGE(SB(1, 1), b3 + HTB);
        WAIT_V(6); BAR; MMA(1, 1, At, B1); BAR;
        k1 = k3;
    }
    WAIT_V(0);
    if (wr == 0) BAR;
    BAR;
    STAMP(own, 2);

    int tz = threadIdx.x; asm volatile("" : "+v"(tz));
    const int lane2 = tz & 63, fr = lane2 & 15, fq = lane2 >> 4;
    float am = 0.f;
#pragma unroll
    for (int a = 0; a < 2; ++a)
#pragma unroll
        for (int c = 0; c < 2; ++c)
#pragma unroll
            for (int m = 0; m < 4; ++m)
#pragma unroll
                for (int n = 0; n < 2; ++n) {
                    const f32x4 v = acc[a][c][m][n];
                    asm("v_max3_f32 %0, |%1|, |%2|, %0" : "+v"(am) : "v"(v[0]), "v"(v[1]));
                    asm("v_max3_f32 %0, |%1|, |%2|, %0" : "+v"(am) : "v"(v[2]), "v"(v[3]));
                }
    LAS unsigned* red = (LAS unsigned*)(lds + LDS_CTL + 64);
    { const unsigned wm = wave_umax(__float_as_uint(am)); if (lane2 == 0) red[wid] = wm; }
    __syncthreads();
    if (tz < NR) {
        const u32x4 r0 = *(LAS u32x4*)red, r1 = *(LAS u32x4*)(red + 4);
        const unsigned m8 = max(max(max(r0.x, r0.y), max(r0.z, r0.w)), max(max(r1.x, r1.y), max(r1.z, r1.w)));
        __hip_atomic_store((gu64*)amax_dst, ((unsigned long long)call << 32) | m8, __ATOMIC_RELAXED, __HIP_MEMORY_SCOPE_SYSTEM);
    }
    STAMP(own, 10);
    float gm;
    {
        gu64* g0 = (gu64*)(own + C_AMAX) + tz;
        const unsigned long long t0 = now();
        unsigned long long a0, a1;
        for (;;) {
            a0 = __hip_atomic_load(g0, __ATOMIC_RELAXED, __HIP_MEMORY_SCOPE_SYSTEM);
            a1 = __hip_atomic_load(g0 + 512, __ATOMIC_RELAXED, __HIP_MEMORY_SCOPE_SYSTEM);
            if (__all((unsigned)(a0 >> 32) >= call && (unsigned)(a1 >> 32) >= call)) break;
            if (give_up(t0, own, call)) break;
            __builtin_amdgcn_s_sleep(1);
        }
        const unsigned wm = wave_umax(max((unsigned)a0, (unsigned)a1));
        if (lane2 == 0) red[8 + wid] = wm;
        __syncthreads();
        const u32x4 r0 = *(LAS u32x4*)(red + 8), r1 = *(LAS u32x4*)(red + 12);
        gm = __uint_as_float(max(max(max(r0.x, r0.y), max(r0.z, r0.w)), max(max(r1.x, r1.y), max(r1.z, r1.w))));
    }
    STAMP(own, 3);
    const bool poison = ld_agent32(own + C_TMO) == call;
    const float inv = gm > 0.f ? 127.0f / gm : 0.f;
    signed char* obase = out_tile + (size_t)(wr * 64 + fr) * NN + wc * 64 + fq * 16;
    constexpr float MAGIC = 12582912.0f;
#pragma unroll
    for (int a = 0; a < 2; ++a)
#pragma unroll
        for (int m = 0; m < 4; ++m) {
            u32x4 o;
#pragma unroll
            for (int c = 0; c < 2; ++c)
#pragma unroll
                for (int n = 0; n < 2; ++n) {
                    const f32x4 v = acc[a][c][m][n];
                    unsigned q[4];
#pragma unroll
                    for (int e = 0; e < 4; ++e)
                        q[e] = __float_as_uint(__builtin_amdgcn_fmed3f(__builtin_fmaf(v[e], inv, MAGIC), MAGIC - 127.0f, MAGIC + 127.0f));
                    const unsigned lo = __builtin_amdgcn_perm(q[1], q[0], 0x0c0c0400u), hi = __builtin_amdgcn_perm(q[3], q[2], 0x04000c0cu);
                    o[c * 2 + n] = lo | hi;
                }
            if (poison) o = (u32x4){0x80808080u, 0x80808080u, 0x80808080u, 0x80808080u};
            st_wt16(obase + (size_t)(a * 128 + m * 16) * NN, o);
        }
    WAIT_V(0);
    __syncthreads();
    if (tz == 0) st_sys32(done_flag, call);
    STAMP(own, 4);
#undef SA
#undef SB
#undef STAGE
#undef LDA
#undef LDB
#undef MMA
}

__global__ __launch_bounds__(NTHR, 2) void gemm_rs_q8_kernel(Params P) {
    extern __shared__ __attribute__((aligned(16))) unsigned char lds_raw[];
    LAS unsigned char* lds = (LAS unsigned char*)lds_raw;
    unsigned callv = 0;
    if ((threadIdx.x & 63) == 0)
        callv = __hip_atomic_fetch_add((gu32*)(P.ws[P.rank] + C_CNT + (size_t)(blockIdx.x * 8 + (threadIdx.x >> 6)) * 16), 1u, __ATOMIC_RELAXED, __HIP_MEMORY_SCOPE_AGENT) + 1u;
    if (blockIdx.x < NSEND) comm(P, blockIdx.x, callv, lds);
    else gemm(P, blockIdx.x - NSEND, (unsigned)__builtin_amdgcn_readfirstlane(callv), lds);
}

}

extern "C" void kernel_launch(int rank, int nranks, ncclComm_t comm,
                              hipStream_t stream,
                              void* const* d_in, const int* in_sizes, int n_in,
                              void* d_out, int out_size,
                              const int* in_dtypes, int out_dtype,
                              void* const* d_out_all,
                              void* const* d_ws_all, size_t ws_bytes) {
    (void)comm; (void)in_dtypes; (void)out_dtype; (void)d_out;
    if (nranks != NR || n_in != 2 || rank < 0 || rank >= NR || in_sizes[0] != MM * KL || in_sizes[1] != KL * NN ||
        out_size != ML * NN || ws_bytes < WS_USED) {
        fprintf(stderr, "kernel_launch: unexpected problem shape\n");
        return;
    }
    Params P;
    P.x = (const unsigned short*)d_in[0];
    P.w = (const unsigned short*)d_in[1];
    for (int p = 0; p < NR; ++p) { P.ws[p] = (unsigned char*)d_ws_all[p]; P.out[p] = (signed char*)d_out_all[p]; }
    P.rank = rank; P.pad = 0;
    HIP_CHECK(hipFuncSetAttribute((const void*)gemm_rs_q8_kernel, hipFuncAttributeMaxDynamicSharedMemorySize, LDS_BYTES));
    hipLaunchKernelGGL(gemm_rs_q8_kernel, dim3(NBLK), dim3(NTHR), LDS_BYTES, stream, P);
    HIP_CHECK(hipGetLastError());
}
```

```cpp
#include <hip/hip_runtime.h>
#include <rccl/rccl.h>
#include <cstdio>
#include <cstdint>

#define HIP_CHECK(expr) do { hipError_t _e = (expr); if (_e != hipSuccess) { \
    fprintf(stderr, "HIP error %s at %s:%d\n", hipGetErrorString(_e), __FILE__, __LINE__); \
    return; } } while (0)

namespace {

#define LAS __attribute__((address_space(3)))
#define GAS __attribute__((address_space(1)))
typedef short bf16x8 __attribute__((ext_vector_type(8)));
typedef short s16x4 __attribute__((ext_vector_type(4)));
typedef float f32x4 __attribute__((ext_vector_type(4)));
typedef unsigned u32x4 __attribute__((ext_vector_type(4)));
typedef unsigned u32x2 __attribute__((ext_vector_type(2)));
typedef GAS unsigned gu32;
typedef GAS unsigned long long gu64;

constexpr int NR = 8;
constexpr int MM = 4096, NN = 16384;
constexpr int KL = 512;
constexpr int NL = NN / NR;
constexpr int ML = MM / NR;
constexpr int NSL = KL / 64;
constexpr int NRB = MM / 256;
constexpr int NCB = NL / 256;
constexpr int NKT = NR * NSL;
constexpr int CHUNK = 32768;
constexpr int NSEND = 128, NGEMM = 128, NBLK = NSEND + NGEMM, NTHR = 512;

constexpr int NPC = NSL * 24 * 4;
constexpr int PK_E = 4096, PK_E2 = 5120, PK_NAR = 5632, PK_WID = 6144, PK_GR = 6400, SLOT = 6528;
constexpr int NAR_CAP = 256, WID_CAP = 64;
constexpr size_t OFF_A = 0;
constexpr size_t OFF_B = (size_t)NR * NSL * NRB * CHUNK;
constexpr size_t OFF_P = OFF_B + (size_t)NR * NSL * NCB * CHUNK;
constexpr size_t OFF_C = OFF_P + (size_t)NR * NPC * SLOT;
constexpr size_t C_FA = OFF_C;
constexpr size_t C_FB = C_FA + NRB * NR * 128;
constexpr size_t C_AMAX = C_FB + NCB * NR * 128;
constexpr size_t C_DONE = C_AMAX + NR * NGEMM * 8;
constexpr size_t C_CNT = C_DONE + NR * 128;
constexpr size_t C_TICK = C_CNT + NBLK * 8 * 16;
constexpr size_t C_TMO = C_TICK + 2 * NR * 128;
constexpr size_t C_STAMP = C_TMO + 128;
constexpr int NSTAMP = 128;
constexpr size_t WS_USED = C_STAMP + NBLK * NSTAMP * 8;
static_assert(OFF_C % 128 == 0 && SLOT % 128 == 0, "alignment");

constexpr int LDS_STAGE = 131072;
constexpr int LDS_CTL = LDS_STAGE;
constexpr int LDS_BYTES = LDS_STAGE + 1024;
constexpr unsigned long long DEADLINE = 40000000ull;

struct Params {
    const unsigned short* x;
    const unsigned short* w;
    unsigned char* ws[NR];
    signed char* out[NR];
    int rank;
    int pad;
};

__device__ __forceinline__ unsigned long long now() { return __builtin_amdgcn_s_memrealtime(); }
__device__ __forceinline__ unsigned ld_sys32(const void* p) { return __hip_atomic_load((gu32*)p, __ATOMIC_RELAXED, __HIP_MEMORY_SCOPE_SYSTEM); }
__device__ __forceinline__ void st_sys32(void* p, unsigned v) { __hip_atomic_store((gu32*)p, v, __ATOMIC_RELAXED, __HIP_MEMORY_SCOPE_SYSTEM); }
__device__ __forceinline__ unsigned ld_agent32(const void* p) { return __hip_atomic_load((gu32*)p, __ATOMIC_RELAXED, __HIP_MEMORY_SCOPE_AGENT); }
__device__ __forceinline__ void st_agent32(void* p, unsigned v) { __hip_atomic_store((gu32*)p, v, __ATOMIC_RELAXED, __HIP_MEMORY_SCOPE_AGENT); }
__device__ __forceinline__ void st_wt16(void* p, u32x4 v) {
    asm volatile("global_store_dwordx4 %0, %1, off sc0 sc1\n\ts_nop 1" :: "v"(p), "v"(v) : "memory");
}
template <class T> __device__ __forceinline__ T* uni(T* p) {
    const unsigned long long v = (unsigned long long)p;
    const unsigned lo = __builtin_amdgcn_readfirstlane((unsigned)v), hi = __builtin_amdgcn_readfirstlane((unsigned)(v >> 32));
    return (T*)(((unsigned long long)hi << 32) | lo);
}
#define WAIT_V(n) asm volatile("s_waitcnt vmcnt(" #n ")" ::: "memory")
#define WAIT_L(n) asm volatile("s_waitcnt lgkmcnt(" #n ")" ::: "memory")
#define BAR __builtin_amdgcn_s_barrier()
#define SCHED __builtin_amdgcn_sched_barrier(0)
#define CFENCE asm volatile("" ::: "memory")

#define GAUX 17
#define STAMP(ws_own, idx) do {} while (0)
#define WSTAMP(ws_own, idx) do {} while (0)

__device__ __forceinline__ unsigned wave_umax(unsigned v) {
    v = max(v, (unsigned)__builtin_amdgcn_update_dpp(0, (int)v, 0x111, 0xf, 0xf, false));
    v = max(v, (unsigned)__builtin_amdgcn_update_dpp(0, (int)v, 0x112, 0xf, 0xf, false));
    v = max(v, (unsigned)__builtin_amdgcn_update_dpp(0, (int)v, 0x114, 0xf, 0xf, false));
    v = max(v, (unsigned)__builtin_amdgcn_update_dpp(0, (int)v, 0x118, 0xf, 0xf, false));
    v = max(v, (unsigned)__builtin_amdgcn_update_dpp(0, (int)v, 0x142, 0xa, 0xf, false));
    v = max(v, (unsigned)__builtin_amdgcn_update_dpp(0, (int)v, 0x143, 0xc, 0xf, false));
    return (unsigned)__builtin_amdgcn_readlane((int)v, 63);
}

__device__ __forceinline__ int rot_of(int d) { return d == 0 ? 0 : ((d - 1) * 16) / 7; }

__device__ __forceinline__ bool give_up(unsigned long long t0, unsigned char* ws_own, unsigned call) {
    if (ld_agent32(ws_own + C_TMO) == call || now() - t0 > DEADLINE) { st_agent32(ws_own + C_TMO, call); return true; }
    return false;
}

#define SEND_CUS 6
#define RAW_TAIL 0
constexpr int WAVE_LDS = 64 * 144 + NAR_CAP * 2 + WID_CAP * 4;
typedef unsigned short u16x2 __attribute__((ext_vector_type(2)));
__device__ __forceinline__ u16x2 pk(unsigned v) { return __builtin_bit_cast(u16x2, v); }
__device__ __forceinline__ unsigned up(u16x2 v) { return __builtin_bit_cast(unsigned, v); }
template <int OFF> __device__ __forceinline__ void st_wt16o(void* p, u32x4 v) {
    asm volatile("global_store_dwordx4 %0, %1, off offset:%2 sc0 sc1\n\ts_nop 1" :: "v"(p), "v"(v), "i"(OFF) : "memory");
}
__device__ __forceinline__ void add_sys32(void* p) { (void)__hip_atomic_fetch_add((gu32*)p, 1u, __ATOMIC_RELAXED, __HIP_MEMORY_SCOPE_SYSTEM); }

struct PieceId { int isB, s, idx, pc; };
__device__ __forceinline__ PieceId piece_of(int q, int rot) {
    const int c = q >> 2, s = c / 24, u = c - 24 * s;
    PieceId id; id.isB = u < 2 * NCB && !(u & 1); id.s = s;
    id.idx = id.isB ? (((u >> 1) + (rot >> 1)) & 7) : (((u < 2 * NCB ? u >> 1 : u - NCB) + rot) & 15);
    id.pc = q & 3; return id;
}
__device__ __forceinline__ unsigned char* image_of(unsigned char* ws, int src, const PieceId& id, int lane, int& stride4) {
    stride4 = id.isB ? 16384 : 4096;
    return id.isB ? ws + OFF_B + (size_t)((src * NSL + id.s) * NCB + id.idx) * CHUNK + id.pc * 4096 + lane * 16
                  : ws + OFF_A + (size_t)((src * NSL + id.s) * NRB + id.idx) * CHUNK + id.pc * 8192 + lane * 16;
}
__device__ __forceinline__ unsigned char* counter_of(unsigned char* ws, int src, const PieceId& id) {
    return (id.isB ? ws + C_FB : ws + C_FA) + (size_t)(id.idx * NR + src) * 128 + id.s * 4;
}
__device__ __forceinline__ void store_image(unsigned char* img, int stride4, const u32x4 (&v)[8]) {
    st_wt16o<0>(img, v[0]); st_wt16o<1024>(img, v[1]); st_wt16o<2048>(img, v[2]); st_wt16o<3072>(img, v[3]);
    unsigned char* i2 = img + stride4;
    st_wt16o<0>(i2, v[4]); st_wt16o<1024>(i2, v[5]); st_wt16o<2048>(i2, v[6]); st_wt16o<3072>(i2, v[7]);
}
__device__ __forceinline__ void issue_piece(const Params& P, int dest, const PieceId& id, int lane, u32x4 (&r)[8]) {
    if (id.isB) {
        const unsigned char* src = (const unsigned char*)P.w + (size_t)(64 * id.s + (lane >> 3)) * (NN * 2) + (size_t)(NL * dest + 256 * id.idx + 64 * id.pc + (lane & 7) * 8) * 2;
#pragma unroll
        for (int it = 0; it < 8; ++it) r[it] = *(const GAS u32x4*)(src + (size_t)(it * 8) * (NN * 2));
    } else {
        const unsigned char* src = (const unsigned char*)P.x + (size_t)(256 * id.idx + 64 * id.pc + (lane & 15)) * (KL * 2) + (64 * id.s + 8 * (lane >> 4)) * 2;
#pragma unroll
        for (int ff = 0; ff < 8; ++ff) r[ff] = *(const GAS u32x4*)(src + (size_t)(16 * (ff >> 1)) * (KL * 2) + (ff & 1) * 64);
    }
}
__device__ __forceinline__ void finish_piece(const PieceId& id, int lane, LAS unsigned char* wl, u32x4 (&v)[8]) {
    if (id.isB) {
#pragma unroll
        for (int it = 0; it < 8; ++it) *(LAS u32x4*)(wl + (it * 8 + (lane >> 3)) * 144 + (lane & 7) * 16) = v[it];
        WAIT_L(0);
        const int fg = lane >> 4, q4 = (lane >> 2) & 3, p4 = lane & 3;
        LAS unsigned char* a0 = wl + (8 * fg + q4) * 144 + p4 * 32;
#pragma unroll
        for (int ff = 0; ff < 8; ++ff) {
            const int hn = ff >> 1, kf = ff & 1;
            LAS unsigned char* a = a0 + (32 * kf) * 144 + hn * 8;
            typedef short v4i16_t __attribute__((ext_vector_type(4)));
            const v4i16_t lo = __builtin_amdgcn_ds_read_tr16_b64_v4i16((LAS v4i16_t*)a);
            const v4i16_t hi = __builtin_amdgcn_ds_read_tr16_b64_v4i16((LAS v4i16_t*)(a + 4 * 144));
            const u32x2 l2 = __builtin_bit_cast(u32x2, lo), h2 = __builtin_bit_cast(u32x2, hi);
            v[ff] = (u32x4){l2.x, l2.y, h2.x, h2.y};
        }
        WAIT_L(0);
    }
}

constexpr unsigned META_RAW = 0x100u;
__device__ __forceinline__ bool pack_piece(const u32x4 (&v)[8], int lane, LAS unsigned char* wl, unsigned char* sl, unsigned& meta) {
    u16x2 mx = pk(0u);
#pragma unroll
    for (int ff = 0; ff < 8; ++ff) {
        *(LAS u32x4*)(wl + ff * 1024 + lane * 16) = v[ff];
#pragma unroll
        for (int i = 0; i < 4; ++i) mx = __builtin_elementwise_max(mx, pk(v[ff][i] & 0x7fff7fffu));
    }
    const unsigned hi = wave_umax(max((unsigned)mx.x, (unsigned)mx.y)) >> 7;
    const unsigned base = hi > 7u ? hi - 7u : 0u, base2 = base * 0x00010001u;
    u32x4 sm[4]; unsigned c12[8], mk[2] = {0u, 0u};
#pragma unroll
    for (int ff = 0; ff < 8; ++ff) {
        unsigned n = 0, t[4];
#pragma unroll
        for (int i = 0; i < 4; ++i) {
            const unsigned dw = v[ff][i];
            const u16x2 ex = pk((dw >> 7) & 0x00ff00ffu);
            n |= up(__builtin_elementwise_sub_sat(ex, pk(base2))) << (3 * i);
            mk[ff >> 2] |= up(__builtin_elementwise_min(__builtin_elementwise_sub_sat(pk(base2), ex), pk(0x00010001u))) << (4 * (ff & 3) + i);
            t[i] = (dw & 0x007f007fu) | ((dw >> 8) & ~0x007f007fu);
        }
        sm[ff >> 1][(ff & 1) * 2] = __builtin_amdgcn_perm(t[1], t[0], 0x06040200u);
        sm[ff >> 1][(ff & 1) * 2 + 1] = __builtin_amdgcn_perm(t[3], t[2], 0x06040200u);
        c12[ff] = n;
    }
    LAS unsigned short* nar = (LAS unsigned short*)(wl + 8192 + 512);
    LAS unsigned* wid = (LAS unsigned*)(wl + 9216);
    unsigned nn = 0, nw = 0;
    while (__ballot((mk[0] | mk[1]) != 0u)) {
        const bool act = (mk[0] | mk[1]) != 0u, g = mk[0] == 0u;
        const unsigned cur = g ? mk[1] : mk[0];
        const unsigned bit = act ? (unsigned)__builtin_ctz(cur) : 0u;
        const unsigned idx = (((g ? 4u : 0u) + ((bit >> 2) & 3u)) * 64u + (unsigned)lane) * 8u + 2u * (bit & 3u) + (bit >> 4);
        const unsigned val = *(LAS unsigned short*)(wl + idx * 2u);
        const unsigned d = base - ((val >> 7) & 0xffu);
        const bool wide = act && d > 15u;
        const unsigned long long bn = __ballot(act && !wide), bw = __ballot(wide);
        const unsigned pos = nn + __builtin_amdgcn_mbcnt_hi((unsigned)(bn >> 32), __builtin_amdgcn_mbcnt_lo((unsigned)bn, 0u));
        if (act && !wide && pos < (unsigned)NAR_CAP) nar[pos] = (unsigned short)((idx << 4) | d);
        nn += (unsigned)__builtin_popcountll(bn);
        if (bw) {
            const unsigned pw = nw + __builtin_amdgcn_mbcnt_hi((unsigned)(bw >> 32), __builtin_amdgcn_mbcnt_lo((unsigned)bw, 0u));
            if (wide && pw < (unsigned)WID_CAP) wid[pw] = (idx << 16) | val;
            nw += (unsigned)__builtin_popcountll(bw);
        }
        if (g) mk[1] = cur & (cur - 1u); else mk[0] = cur & (cur - 1u);
        if (nn > (unsigned)NAR_CAP || nw > (unsigned)WID_CAP) return false;
    }
    unsigned char* a = sl + lane * 16;
    st_wt16o<0>(a, sm[0]); st_wt16o<1024>(a, sm[1]); st_wt16o<2048>(a, sm[2]); st_wt16o<3072>(a, sm[3]);
    unsigned w[6];
#pragma unroll
    for (int g = 0; g < 2; ++g) {
        const u16x2 ca = pk(c12[4 * g]), cb = pk(c12[4 * g + 1]), cc = pk(c12[4 * g + 2]), cd = pk(c12[4 * g + 3]);
        w[3 * g] = up(ca | (cb << (unsigned short)12));
        w[3 * g + 1] = up((cb >> (unsigned short)4) | (cc << (unsigned short)8));
        w[3 * g + 2] = up((cc >> (unsigned short)8) | (cd << (unsigned short)4));
    }
    st_wt16o<0>(a + PK_E, (u32x4){w[0], w[1], w[2], w[3]});
    *(LAS u32x2*)(wl + 8192 + lane * 8) = (u32x2){w[4], w[5]};
    WAIT_L(0);
    if (16u * lane < 512u + 2u * nn) st_wt16o<0>(a + PK_E2, *(volatile LAS u32x4*)(wl + 8192 + lane * 16));
    if (4u * lane < nw) st_wt16o<0>(a + PK_WID, *(volatile LAS u32x4*)(wl + 9216 + lane * 16));
    meta = base | (nn << 9) | (nw << 18);
    return true;
}
__device__ __forceinline__ void unpack_piece(const unsigned char* sl, unsigned meta, int lane, LAS unsigned char* wl, u32x4 (&v)[8]) {
    u32x4 k0, k1, k2, k3, e4; u32x2 e2; unsigned na, nb, wd;
    const unsigned char* a = sl + lane * 16;
    asm volatile("global_load_dwordx4 %0, %9, off sc0 sc1\n\t"
                 "global_load_dwordx4 %1, %9, off offset:1024 sc0 sc1\n\t"
                 "global_load_dwordx4 %2, %9, off offset:2048 sc0 sc1\n\t"
                 "global_load_dwordx4 %3, %9, off offset:3072 sc0 sc1\n\t"
                 "global_load_dwordx4 %4, %10, off sc0 sc1\n\t"
                 "global_load_dwordx2 %5, %11, off sc0 sc1\n\t"
                 "global_load_dword %6, %12, off sc0 sc1\n\t"
                 "global_load_dword %7, %12, off offset:256 sc0 sc1\n\t"
                 "global_load_dword %8, %12, off offset:512 sc0 sc1\n\t"
                 "s_waitcnt vmcnt(0)"
                 : "=&v"(k0), "=&v"(k1), "=&v"(k2), "=&v"(k3), "=&v"(e4), "=&v"(e2), "=&v"(na), "=&v"(nb), "=&v"(wd)
                 : "v"(a), "v"(a + PK_E), "v"(sl + PK_E2 + lane * 8), "v"(sl + PK_NAR + lane * 4) : "memory");
    static_assert(PK_WID == PK_NAR + 512, "list offsets");
    const u32x4 sm[4] = {k0, k1, k2, k3};
    const unsigned w[6] = {e4.x, e4.y, e4.z, e4.w, e2.x, e2.y};
    unsigned c12[8];
#pragma unroll
    for (int g = 0; g < 2; ++g) {
        const u16x2 w0 = pk(w[3 * g]), w1 = pk(w[3 * g + 1]), w2 = pk(w[3 * g + 2]);
        c12[4 * g] = up(w0);
        c12[4 * g + 1] = up((w0 >> (unsigned short)12) | (w1 << (unsigned short)4));
        c12[4 * g + 2] = up((w1 >> (unsigned short)8) | (w2 << (unsigned short)8));
        c12[4 * g + 3] = up(w2 >> (unsigned short)4);
    }
    const unsigned base7 = ((meta & 0xffu) << 7) * 0x00010001u;
#pragma unroll
    for (int ff = 0; ff < 8; ++ff) {
        u32x4 o;
#pragma unroll
        for (int i = 0; i < 4; ++i) {
            const unsigned s4 = sm[ff >> 1][(ff & 1) * 2 + (i >> 1)];
            const unsigned u = __builtin_amdgcn_perm(s4, s4, (i & 1) ? 0x03030202u : 0x01010000u);
            const unsigned e = (((c12[ff] >> (3 * i)) & 0x00070007u) << 7) + base7;
            o[i] = (u & 0x807f807fu) | e;
        }
        *(LAS u32x4*)(wl + ff * 1024 + lane * 16) = o;
    }
    const unsigned nn = (meta >> 9) & 0x1ffu, nw = meta >> 18;
#pragma unroll
    for (int r = 0; r < 4; ++r) {
        const unsigned e = (unsigned)((r >> 1) * 128 + 2 * lane + (r & 1));
        if (e < nn) {
            const unsigned ent = ((r >> 1 ? nb : na) >> (16 * (r & 1))) & 0xffffu, idx = ent >> 4;
            (void)__hip_atomic_fetch_sub((LAS unsigned*)(wl + (idx >> 1) * 4), ((ent & 15u) << 7) << (16 * (idx & 1u)), __ATOMIC_RELAXED, __HIP_MEMORY_SCOPE_WORKGROUP);
        }
    }
    if ((unsigned)lane < nw) *(LAS unsigned short*)(wl + ((wd >> 16) & 4095u) * 2) = (unsigned short)wd;
    WAIT_L(0);
#pragma unroll
    for (int ff = 0; ff < 8; ++ff) v[ff] = *(LAS u32x4*)(wl + ff * 1024 + lane * 16);
}

constexpr int NUCU = 16 - SEND_CUS;
constexpr int NUW = NUCU * 8 + 18;
constexpr int NSELF = 128 + 7 * NUCU * 8;

__device__ __forceinline__ void comm(const Params& P, int b, unsigned callv, LAS unsigned char* lds) {
    const int tid = threadIdx.x, wid = __builtin_amdgcn_readfirstlane(tid >> 6), lane = tid & 63;
    const int rank = P.rank;
    const int xcd = b & 7, slot = b >> 3;
    unsigned char* own = P.ws[rank];
    LAS unsigned char* wl = lds + wid * WAVE_LDS;
    STAMP(own, 0);
    if (xcd != 0 && slot < SEND_CUS) {
        const int d = xcd, dest = (rank + d) & 7, rot = rot_of(d);
        constexpr int nw = SEND_CUS * 8;
        unsigned char* wsd = P.ws[dest];
        gu32* tick = (gu32*)(own + C_TICK + (size_t)d * 128);
        u32x4 nxt[8];
        int q = slot * 8 + wid;
        issue_piece(P, dest, piece_of(q, rot), lane, nxt);
        const unsigned call = __builtin_amdgcn_readfirstlane(callv);
        const unsigned tbase = (call - 1u) * (unsigned)NPC - (unsigned)nw;
        unsigned tk = 0;
        if (lane == 0) tk = __hip_atomic_fetch_add(tick, 1u, __ATOMIC_RELAXED, __HIP_MEMORY_SCOPE_AGENT);
        unsigned char* pub = nullptr; unsigned long long pubv = 0; bool pubdirect = false;
        for (;;) {
            const PieceId id = piece_of(q, rot);
            u32x4 v[8];
            asm volatile("s_waitcnt vmcnt(0)" : "+v"(nxt[0]), "+v"(nxt[1]), "+v"(nxt[2]), "+v"(nxt[3]), "+v"(nxt[4]), "+v"(nxt[5]), "+v"(nxt[6]), "+v"(nxt[7]), "+v"(tk) :: "memory");
            if (pub && lane == 0) {
                if (pubdirect) add_sys32(pub);
                else __hip_atomic_store((gu64*)pub, pubv, __ATOMIC_RELAXED, __HIP_MEMORY_SCOPE_SYSTEM);
            }
#pragma unroll
            for (int ff = 0; ff < 8; ++ff) v[ff] = nxt[ff];
            const unsigned qn = (unsigned)__builtin_amdgcn_readfirstlane(tk) - tbase;
            const bool more = qn < (unsigned)NPC;
            if (more) {
                issue_piece(P, dest, piece_of((int)qn, rot), lane, nxt);
                if (lane == 0) tk = __hip_atomic_fetch_add(tick, 1u, __ATOMIC_RELAXED, __HIP_MEMORY_SCOPE_AGENT);
            }
            finish_piece(id, lane, wl, v);
            const bool direct = q >= NPC - RAW_TAIL;
            unsigned meta = META_RAW;
            unsigned char* sl = wsd + OFF_P + (size_t)(rank * NPC + q) * SLOT;
            if (direct || !pack_piece(v, lane, wl, sl, meta)) { meta = META_RAW; int st4; unsigned char* img = image_of(wsd, rank, id, lane, st4); store_image(img, st4, v); }
            pubdirect = direct;
            pub = direct ? counter_of(wsd, rank, id) : sl + PK_GR;
            pubv = ((unsigned long long)call << 32) | meta;
            if (!more) break;
            q = (int)qn;
        }
        WAIT_V(0);
        if (pub && lane == 0) {
            if (pubdirect) add_sys32(pub);
            else __hip_atomic_store((gu64*)pub, pubv, __ATOMIC_RELAXED, __HIP_MEMORY_SCOPE_SYSTEM);
        }
        STAMP(own, 1); WSTAMP(own, 8);
    } else {
        const int g0 = xcd == 0 ? slot * 8 + wid : 128 + ((xcd - 1) * NUCU + slot - SEND_CUS) * 8 + wid;
        for (int q = g0; q < NPC; q += NSELF) {
            const PieceId id = piece_of(q, 0);
            u32x4 v[8];
            issue_piece(P, rank, id, lane, v);
            finish_piece(id, lane, wl, v);
            int st4; unsigned char* img = image_of(own, rank, id, lane, st4);
            store_image(img, st4, v);
            WAIT_V(0);
            if (lane == 0) add_sys32(counter_of(own, rank, id));
        }
        STAMP(own, 11);
        const unsigned call = __builtin_amdgcn_readfirstlane(callv);
        const int d = xcd != 0 ? xcd : 1 + g0 % 7;
        const int first = xcd != 0 ? (slot - SEND_CUS) * 8 + wid : NUCU * 8 + g0 / 7;
        const int src = (rank - d) & 7, rot = rot_of(d);
        constexpr int NUP = NPC - RAW_TAIL;
        gu32* tick = (gu32*)(own + C_TICK + (size_t)(NR + d) * 128);
        const unsigned tbase = (call - 1u) * (unsigned)NUP - (unsigned)NUW;
        unsigned tk = 0;
        for (int q = first < NUW ? first : NUP; q < NUP;) {
            if (lane == 0) tk = __hip_atomic_fetch_add(tick, 1u, __ATOMIC_RELAXED, __HIP_MEMORY_SCOPE_AGENT);
            const PieceId id = piece_of(q, rot);
            const unsigned char* sl = own + OFF_P + (size_t)(src * NPC + q) * SLOT;
            unsigned meta;
            {
                const unsigned long long t0 = now();
                for (;;) {
                    const unsigned long long g = __hip_atomic_load((gu64*)(sl + PK_GR), __ATOMIC_RELAXED, __HIP_MEMORY_SCOPE_SYSTEM);
                    const unsigned tag = __builtin_amdgcn_readfirstlane((unsigned)(g >> 32));
                    meta = __builtin_amdgcn_readfirstlane((unsigned)g);
                    if (tag >= call) break;
                    if (give_up(t0, own, call)) { meta = META_RAW; break; }
                    __builtin_amdgcn_s_sleep(2);
                }
            }
            if (!(meta & META_RAW)) {
                u32x4 v[8];
                unpack_piece(sl, meta, lane, wl, v);
                int st4; unsigned char* img = image_of(own, src, id, lane, st4);
                store_image(img, st4, v);
                WAIT_V(0);
            }
            if (lane == 0) add_sys32(counter_of(own, src, id));
            const unsigned qn = (unsigned)__builtin_amdgcn_readfirstlane(tk) - tbase;
            q = qn < (unsigned)NUP ? (int)qn : NUP;
        }
        STAMP(own, 1); WSTAMP(own, 8);
    }
    if (b == 0) {
        const unsigned call = __builtin_amdgcn_readfirstlane(callv);
        if (wid < 2) {
            const unsigned char* f = own + C_DONE + (size_t)(tid >> 4) * 128 + (tid & 15) * 4;
            const unsigned long long t0 = now();
            bool bad = false;
            while (!__all(ld_sys32(f) >= call)) {
                if (give_up(t0, own, call)) { bad = true; break; }
                __builtin_amdgcn_s_sleep(2);
            }
            if (bad && lane == 0) P.out[rank][rank * NL] = (signed char)-128;
        }
        STAMP(own, 2);
    }
}

__device__ __forceinline__ void gemm(const Params& P, int b, unsigned call, LAS unsigned char* lds) {
    const int tid = threadIdx.x, wid = __builtin_amdgcn_readfirstlane(tid >> 6), lane = tid & 63, wr = wid >> 2, wc = wid & 3;
    const int rank = P.rank;
    const int xcd = b & 7, slot = b >> 3;
    const int ti = 2 * ((rank + xcd) & 7) + (slot >> 3), tj = slot & 7;
    unsigned char* own = P.ws[rank];
    STAMP(own, 0);
    const unsigned char* abase = uni(own + OFF_A + (size_t)ti * CHUNK);
    const unsigned char* bbase = uni(own + OFF_B + (size_t)tj * CHUNK);
    auto aptr = [&](int k) -> const unsigned char* { return abase + ((size_t)(unsigned)k << 19); };
    auto bptr = [&](int k) -> const unsigned char* { return bbase + ((size_t)(unsigned)k << 18); };
    static_assert(NRB * CHUNK == (1 << 19) && NCB * CHUNK == (1 << 18), "chunk strides");

    LAS unsigned* ctl = (LAS unsigned*)(lds + LDS_CTL);
    const unsigned ctl_addr = (unsigned)(size_t)ctl;
    const unsigned char* fbaseA = uni(own + C_FA + (size_t)ti * NR * 128);
    const unsigned char* fbaseB = uni(own + C_FB + (size_t)tj * NR * 128);
    const unsigned foff = (unsigned)(lane >> 3) * 128u + (unsigned)(lane & 7) * 4u;
    const unsigned target = 4u * call;
    unsigned long long known = 0, taken = 0;
    auto pick = [&]() -> int {
        if ((known & ~taken) == 0) {
            if (taken == ~0ull) return 0;
            if (wid == 0) {
                const unsigned long long t0 = now();
                unsigned long long m;
                for (;;) {
                    unsigned va, vb;
                    asm volatile("global_load_dword %0, %2, %3 sc0 sc1\n\tglobal_load_dword %1, %2, %4 sc0 sc1\n\ts_waitcnt vmcnt(0)"
                                 : "=&v"(va), "=&v"(vb) : "v"(foff), "s"(fbaseA), "s"(fbaseB) : "memory");
                    m = __ballot(va >= target && vb >= target);
                    if (m & ~taken) break;
                    if (give_up(t0, own, call)) { m = ~0ull; break; }
                    __builtin_amdgcn_s_sleep(1);
                }
                const u32x2 mv = {(unsigned)m, (unsigned)(m >> 32)};
                asm volatile("ds_write_b64 %0, %1\n\ts_waitcnt lgkmcnt(0)" :: "v"(ctl_addr), "v"(mv) : "memory");
            }
            CFENCE; BAR; CFENCE;
            u32x2 r;
            asm volatile("ds_read_b64 %0, %1\n\ts_waitcnt lgkmcnt(0)" : "=v"(r) : "v"(ctl_addr) : "memory");
            known |= ((unsigned long long)(unsigned)__builtin_amdgcn_readfirstlane(r.y) << 32) | (unsigned)__builtin_amdgcn_readfirstlane(r.x);
        }
        const int k = __builtin_ctzll(known & ~taken);
        taken |= 1ull << k;
        return k;
    };
    unsigned char* amax_dst = P.ws[lane & 7] + C_AMAX + (size_t)(rank * NGEMM + b) * 8;
    asm volatile("" : "+v"(amax_dst));
    const int dest = ti >> 1;
    signed char* out_tile = uni(P.out[dest] + (size_t)((ti & 1) * 256) * NN + (size_t)rank * NL + tj * 256);
    unsigned char* done_flag = uni(P.ws[dest] + C_DONE + (size_t)rank * 128 + ((ti & 1) * 8 + tj) * 4);

    f32x4 acc[2][2][4][2];
#pragma unroll
    for (int a = 0; a < 2; ++a)
#pragma unroll
        for (int c = 0; c < 2; ++c)
#pragma unroll
            for (int m = 0; m < 4; ++m)
#pragma unroll
                for (int n = 0; n < 2; ++n) acc[a][c][m][n] = (f32x4){0.f, 0.f, 0.f, 0.f};
    bf16x8 At[4][2], B0[2][2], B1[2][2];
    unsigned voff[2] = {(unsigned)tid * 16u, (unsigned)tid * 16u + 8192u};
    asm volatile("" : "+v"(voff[0]), "+v"(voff[1]));
    const unsigned ldsw = (unsigned)wid * 1024u;
    const int aoff = wr * 8192 + lane * 16, boff = wc * 4096 + lane * 16;
    constexpr int HTB = 16384;
#define SA(b_, h_) (((b_) * 2 + (h_)) * HTB)
#define SB(b_, h_) ((4 + (b_) * 2 + (h_)) * HTB)
#define STAGE(bufoff, gbase) do { _Pragma("unroll") for (int _i = 0; _i < 2; ++_i) \
        __builtin_amdgcn_global_load_lds((const unsigned*)((const char*)(gbase) + voff[_i]), (LAS unsigned*)(lds + (bufoff) + ldsw + _i * 8192), 16, 0, GAUX); } while (0)
#define LDA(dst, b_, h_) do { _Pragma("unroll") for (int m = 0; m < 4; ++m) _Pragma("unroll") for (int k = 0; k < 2; ++k) dst[m][k] = *(const LAS bf16x8*)(lds + SA(b_, h_) + aoff + m * 2048 + k * 1024); } while (0)
#define LDB(dst, b_, h_) do { _Pragma("unroll") for (int n = 0; n < 2; ++n) _Pragma("unroll") for (int k = 0; k < 2; ++k) dst[n][k] = *(const LAS bf16x8*)(lds + SB(b_, h_) + boff + n * 2048 + k * 1024); } while (0)
#define MMA(ai, bj, At_, Bt_) do { __builtin_amdgcn_s_setprio(1); _Pragma("unroll") for (int m = 0; m < 4; ++m) _Pragma("unroll") for (int n = 0; n < 2; ++n) _Pragma("unroll") for (int k = 0; k < 2; ++k) \
        acc[ai][bj][m][n] = __builtin_amdgcn_mfma_f32_16x16x32_bf16(Bt_[n][k], At_[m][k], acc[ai][bj][m][n], 0, 0, 0); __builtin_amdgcn_s_setprio(0); } while (0)

    {
        const int k0 = pick();
        const unsigned char* a0 = aptr(k0); const unsigned char* b0 = bptr(k0);
        STAGE(SB(0, 0), b0); STAGE(SB(0, 1), b0 + HTB); STAGE(SA(0, 0), a0); STAGE(SA(0, 1), a0 + HTB);
    }
    int k1 = pick();
    if (wr == 1) BAR;
    WAIT_V(2); BAR;
    {
        const unsigned char* a1 = aptr(k1); const unsigned char* b1 = bptr(k1);
        STAGE(SB(1, 0), b1); STAGE(SA(1, 0), a1); STAGE(SB(1, 1), b1 + HTB);
    }
    WAIT_V(6); BAR;
    STAMP(own, 1);
#define LOOK_ISSUE() do { if (look) asm volatile("global_load_dword %0, %2, %3 sc0 sc1\n\tglobal_load_dword %1, %2, %4 sc0 sc1" : "=&v"(pa), "=&v"(pb) : "v"(foff), "s"(fbaseA), "s"(fbaseB) : "memory"); } while (0)
#define LOOK_PUBLISH(OFF) do { if (wid == 0) { unsigned long long m_ = known; \
            if (look) { asm volatile("" : "+v"(pa), "+v"(pb)); m_ |= __ballot(pa >= target && pb >= target); } \
            const u32x2 mv_ = {(unsigned)m_, (unsigned)(m_ >> 32)}; \
            asm volatile("ds_write_b64 %0, %1 offset:" #OFF :: "v"(ctl_addr), "v"(mv_) : "memory"); } } while (0)
#define LOOK_READ(OFF) asm volatile("ds_read_b64 %0, %1 offset:" #OFF : "=v"(rn) : "v"(ctl_addr) : "memory")
#define LOOK_TAKE() do { asm volatile("" : "+v"(rn)); known |= ((unsigned long long)(unsigned)__builtin_amdgcn_readfirstlane(rn.y) << 32) | (unsigned)__builtin_amdgcn_readfirstlane(rn.x); } while (0)
    for (int t = 0; t < NKT; t += 2) {
        const unsigned char* a1 = aptr(k1);
        unsigned pa = 0, pb = 0;
        const bool look = wid == 0 && known != ~0ull;
        u32x2 rn;
        LOOK_ISSUE();
        LDB(B0, 0, 0); LDB(B1, 0, 1); SCHED; LDA(At, 0, 0); STAGE(SA(1, 1), a1 + HTB);
        WAIT_V(8); WAIT_L(0); BAR; MMA(0, 0, At, B0); MMA(0, 1, At, B1); BAR; SCHED;
        const int k2 = pick();
        const unsigned char* a2 = aptr(k2); const unsigned char* b2 = bptr(k2);
        LDA(At, 0, 1); STAGE(SB(0, 0), b2); STAGE(SB(0, 1), b2 + HTB); STAGE(SA(0, 0), a2);
        WAIT_V(8); LOOK_PUBLISH(8);
        WAIT_L(0); BAR; MMA(1, 0, At, B0); MMA(1, 1, At, B1); BAR; SCHED;
        LOOK_READ(8);
        LDB(B0, 1, 0); LDB(B1, 1, 1); SCHED; LDA(At, 1, 0); STAGE(SA(0, 1), a2 + HTB);
        WAIT_V(8); WAIT_L(0); LOOK_TAKE();
        BAR; MMA(0, 0, At, B0); MMA(0, 1, At, B1); BAR; SCHED;
        const int k3 = pick();
        const unsigned char* a3 = aptr(k3); const unsigned char* b3 = bptr(k3);
        LDA(At, 1, 1); STAGE(SB(1, 0), b3); STAGE(SB(1, 1), b3 + HTB); STAGE(SA(1, 0), a3);
        WAIT_V(8); WAIT_L(0); BAR; MMA(1, 0, At, B0); MMA(1, 1, At, B1); BAR; SCHED;
        k1 = k3;
    }
#undef LOOK_ISSUE
#undef LOOK_PUBLISH
#undef LOOK_READ
#undef LOOK_TAKE
    if (wr == 0) BAR;
#define LSTAMP(x) do {} while (0)

    int tz = threadIdx.x; asm volatile("" : "+v"(tz));
    const int lane2 = tz & 63, fr = lane2 & 15, fq = lane2 >> 4;
    float am;
    asm volatile("s_nop 15\n\tv_mov_b32 %0, 0" : "=v"(am));
#pragma unroll
    for (int a = 0; a < 2; ++a)
#pragma unroll
        for (int c = 0; c < 2; ++c)
#pragma unroll
            for (int m = 0; m < 4; ++m)
#pragma unroll
                for (int n = 0; n < 2; ++n) {
                    const f32x4 v = acc[a][c][m][n];
                    asm("v_max3_f32 %0, |%1|, |%2|, %0" : "+v"(am) : "v"(v[0]), "v"(v[1]));
                    asm("v_max3_f32 %0, |%1|, |%2|, %0" : "+v"(am) : "v"(v[2]), "v"(v[3]));
                }
    LAS unsigned* red = (LAS unsigned*)(lds + LDS_CTL + 64);
    { const unsigned wm = wave_umax(__float_as_uint(am)); if (lane2 == 0) red[wid] = wm; }
    WAIT_L(0); BAR; CFENCE;
    if (tz < NR) {
        const u32x4 r0 = *(LAS u32x4*)red, r1 = *(LAS u32x4*)(red + 4);
        const unsigned m8 = max(max(max(r0.x, r0.y), max(r0.z, r0.w)), max(max(r1.x, r1.y), max(r1.z, r1.w)));
        __hip_atomic_store((gu64*)amax_dst, ((unsigned long long)call << 32) | m8, __ATOMIC_RELAXED, __HIP_MEMORY_SCOPE_SYSTEM);
    }
    LSTAMP(ts10);
    float gm; bool poison;
    {
        gu64* g0 = (gu64*)(own + C_AMAX) + tz;
        const unsigned long long t0 = now();
        unsigned long long a0, a1;
        for (;;) {
            a0 = __hip_atomic_load(g0, __ATOMIC_RELAXED, __HIP_MEMORY_SCOPE_SYSTEM);
            a1 = __hip_atomic_load(g0 + 512, __ATOMIC_RELAXED, __HIP_MEMORY_SCOPE_SYSTEM);
            if (__all((unsigned)(a0 >> 32) >= call && (unsigned)(a1 >> 32) >= call)) break;
            if (give_up(t0, own, call)) break;
        }
        LSTAMP(ts12);
        const unsigned tmo = ld_agent32(own + C_TMO);
        const unsigned wm = wave_umax(max((unsigned)a0, (unsigned)a1));
        if (lane2 == 0) { red[8 + wid] = wm; red[16 + wid] = tmo == call; }
        WAIT_L(0); BAR; CFENCE;
        const u32x4 r0 = *(LAS u32x4*)(red + 8), r1 = *(LAS u32x4*)(red + 12), p0 = *(LAS u32x4*)(red + 16), p1 = *(LAS u32x4*)(red + 20);
        gm = __uint_as_float(max(max(max(r0.x, r0.y), max(r0.z, r0.w)), max(max(r1.x, r1.y), max(r1.z, r1.w))));
        poison = ((p0.x | p0.y | p0.z | p0.w) | (p1.x | p1.y | p1.z | p1.w)) != 0u;
    }
    LSTAMP(ts3);
    const float inv = gm > 0.f ? 127.0f / gm : 0.f;
    constexpr float MAGIC = 12582912.0f;
    signed char* obase = out_tile + (size_t)(wr * 64 + fr) * NN + wc * 64 + fq * 16;
#pragma unroll
    for (int a = 0; a < 2; ++a)
#pragma unroll
        for (int m = 0; m < 4; ++m) {
            u32x4 o;
#pragma unroll
            for (int c = 0; c < 2; ++c)
#pragma unroll
                for (int n = 0; n < 2; ++n) {
                    const f32x4 v = acc[a][c][m][n];
                    unsigned q[4];
#pragma unroll
                    for (int e = 0; e < 4; ++e)
                        q[e] = __float_as_uint(__builtin_amdgcn_fmed3f(__builtin_fmaf(v[e], inv, MAGIC), MAGIC - 127.0f, MAGIC + 127.0f));
                    const unsigned lo = __builtin_amdgcn_perm(q[1], q[0], 0x0c0c0400u), hi = __builtin_amdgcn_perm(q[3], q[2], 0x04000c0cu);
                    o[c * 2 + n] = lo | hi;
                }
            if (poison) o = (u32x4){0x80808080u, 0x80808080u, 0x80808080u, 0x80808080u};
            st_wt16(obase + (size_t)(a * 128 + m * 16) * NN, o);
        }
    LSTAMP(ts13);
    WAIT_V(0);
    __syncthreads();
    if (tz == 0) st_sys32(done_flag, call);
    STAMP(own, 4);
#undef SA
#undef SB
#undef STAGE
#undef LDA
#undef LDB
#undef MMA
}

__global__ __launch_bounds__(NTHR, 2) void gemm_rs_q8_kernel(Params P) {
    extern __shared__ __attribute__((aligned(16))) unsigned char lds_raw[];
    LAS unsigned char* lds = (LAS unsigned char*)lds_raw;
    unsigned callv = 0;
    if ((threadIdx.x & 63) == 0)
        callv = __hip_atomic_fetch_add((gu32*)(P.ws[P.rank] + C_CNT + (size_t)(blockIdx.x * 8 + (threadIdx.x >> 6)) * 16), 1u, __ATOMIC_RELAXED, __HIP_MEMORY_SCOPE_AGENT) + 1u;
    if (blockIdx.x < NSEND) comm(P, blockIdx.x, callv, lds);
    else gemm(P, blockIdx.x - NSEND, (unsigned)__builtin_amdgcn_readfirstlane(callv), lds);
}

}

extern "C" void kernel_launch(int rank, int nranks, ncclComm_t comm,
                              hipStream_t stream,
                              void* const* d_in, const int* in_sizes, int n_in,
                              void* d_out, int out_size,
                              const int* in_dtypes, int out_dtype,
                              void* const* d_out_all,
                              void* const* d_ws_all, size_t ws_bytes) {
    (void)comm; (void)in_dtypes; (void)out_dtype; (void)d_out;
    if (nranks != NR || n_in != 2 || rank < 0 || rank >= NR || in_sizes[0] != MM * KL || in_sizes[1] != KL * NN ||
        out_size != ML * NN || ws_bytes < WS_USED) {
        fprintf(stderr, "kernel_launch: unexpected problem shape\n");
        return;
    }
    Params P;
    P.x = (const unsigned short*)d_in[0];
    P.w = (const unsigned short*)d_in[1];
    for (int p = 0; p < NR; ++p) { P.ws[p] = (unsigned char*)d_ws_all[p]; P.out[p] = (signed char*)d_out_all[p]; }
    P.rank = rank; P.pad = 0;
    HIP_CHECK(hipFuncSetAttribute((const void*)gemm_rs_q8_kernel, hipFuncAttributeMaxDynamicSharedMemorySize, LDS_BYTES));
    hipLaunchKernelGGL(gemm_rs_q8_kernel, dim3(NBLK), dim3(NTHR), LDS_BYTES, stream, P);
    HIP_CHECK(hipGetLastError());
}
```

```cpp
#include <hip/hip_runtime.h>
#include <rccl/rccl.h>
#include <cstdio>
#include <cstdint>

#define HIP_CHECK(expr) do { hipError_t _e = (expr); if (_e != hipSuccess) { \
    fprintf(stderr, "HIP error %s at %s:%d\n", hipGetErrorString(_e), __FILE__, __LINE__); \
    return; } } while (0)

namespace {

#define LAS __attribute__((address_space(3)))
#define GAS __attribute__((address_space(1)))
typedef short bf16x8 __attribute__((ext_vector_type(8)));
typedef short s16x4 __attribute__((ext_vector_type(4)));
typedef float f32x4 __attribute__((ext_vector_type(4)));
typedef unsigned u32x4 __attribute__((ext_vector_type(4)));
typedef unsigned u32x2 __attribute__((ext_vector_type(2)));
typedef GAS unsigned gu32;
typedef GAS unsigned long long gu64;

constexpr int NR = 8;
constexpr int MM = 4096, NN = 16384;
constexpr int KL = 512;
constexpr int NL = NN / NR;
constexpr int ML = MM / NR;
constexpr int NSL = KL / 64;
constexpr int NRB = MM / 256;
constexpr int NCB = NL / 256;
constexpr int NKT = NR * NSL;
constexpr int CHUNK = 32768;
constexpr int NSEND = 128, NGEMM = 128, NBLK = NSEND + NGEMM, NTHR = 512;

constexpr int NPC = NSL * 24 * 4;
constexpr int PK_P = 4096, PK_T = 5120, PK_WID = 7168, PK_GR = 7424, SLOT = 7552;
constexpr int TAIL_CAP = PK_WID - PK_T, NAR_CAP = 256, WID_CAP = 64;
constexpr size_t OFF_A = 0;
constexpr size_t OFF_B = (size_t)NR * NSL * NRB * CHUNK;
constexpr size_t OFF_P = OFF_B + (size_t)NR * NSL * NCB * CHUNK;
constexpr size_t OFF_C = OFF_P + (size_t)NR * NPC * SLOT;
constexpr size_t C_FA = OFF_C;
constexpr size_t C_FB = C_FA + NRB * NR * 128;
constexpr size_t C_AMAX = C_FB + NCB * NR * 128;
constexpr size_t C_DONE = C_AMAX + NR * NGEMM * 8;
constexpr size_t C_CNT = C_DONE + NR * 128;
constexpr size_t C_TICK = C_CNT + NBLK * 8 * 16;
constexpr size_t C_TMO = C_TICK + 2 * NR * 128;
constexpr size_t WS_USED = C_TMO + 128;
static_assert(OFF_C % 128 == 0 && SLOT % 128 == 0, "alignment");

constexpr int LDS_STAGE = 131072;
constexpr int LDS_CTL = LDS_STAGE;
constexpr int LDS_BYTES = LDS_STAGE + 1024;
constexpr unsigned long long DEADLINE = 40000000ull;

struct Params {
    const unsigned short* x;
    const unsigned short* w;
    unsigned char* ws[NR];
    signed char* out[NR];
    int rank;
    int pad;
};

__device__ __forceinline__ unsigned long long now() { return __builtin_amdgcn_s_memrealtime(); }
__device__ __forceinline__ unsigned ld_sys32(const void* p) { return __hip_atomic_load((gu32*)p, __ATOMIC_RELAXED, __HIP_MEMORY_SCOPE_SYSTEM); }
__device__ __forceinline__ void st_sys32(void* p, unsigned v) { __hip_atomic_store((gu32*)p, v, __ATOMIC_RELAXED, __HIP_MEMORY_SCOPE_SYSTEM); }
__device__ __forceinline__ unsigned ld_agent32(const void* p) { return __hip_atomic_load((gu32*)p, __ATOMIC_RELAXED, __HIP_MEMORY_SCOPE_AGENT); }
__device__ __forceinline__ void st_agent32(void* p, unsigned v) { __hip_atomic_store((gu32*)p, v, __ATOMIC_RELAXED, __HIP_MEMORY_SCOPE_AGENT); }
__device__ __forceinline__ void st_wt16(void* p, u32x4 v) {
    asm volatile("global_store_dwordx4 %0, %1, off sc0 sc1\n\ts_nop 1" :: "v"(p), "v"(v) : "memory");
}
template <class T> __device__ __forceinline__ T* uni(T* p) {
    const unsigned long long v = (unsigned long long)p;
    const unsigned lo = __builtin_amdgcn_readfirstlane((unsigned)v), hi = __builtin_amdgcn_readfirstlane((unsigned)(v >> 32));
    return (T*)(((unsigned long long)hi << 32) | lo);
}
#define WAIT_V(n) asm volatile("s_waitcnt vmcnt(" #n ")" ::: "memory")
#define WAIT_L(n) asm volatile("s_waitcnt lgkmcnt(" #n ")" ::: "memory")
#define BAR __builtin_amdgcn_s_barrier()
#define SCHED __builtin_amdgcn_sched_barrier(0)
#define CFENCE asm volatile("" ::: "memory")


__device__ __forceinline__ unsigned wave_umax(unsigned v) {
    v = max(v, (unsigned)__builtin_amdgcn_update_dpp(0, (int)v, 0x111, 0xf, 0xf, false));
    v = max(v, (unsigned)__builtin_amdgcn_update_dpp(0, (int)v, 0x112, 0xf, 0xf, false));
    v = max(v, (unsigned)__builtin_amdgcn_update_dpp(0, (int)v, 0x114, 0xf, 0xf, false));
    v = max(v, (unsigned)__builtin_amdgcn_update_dpp(0, (int)v, 0x118, 0xf, 0xf, false));
    v = max(v, (unsigned)__builtin_amdgcn_update_dpp(0, (int)v, 0x142, 0xa, 0xf, false));
    v = max(v, (unsigned)__builtin_amdgcn_update_dpp(0, (int)v, 0x143, 0xc, 0xf, false));
    return (unsigned)__builtin_amdgcn_readlane((int)v, 63);
}

__device__ __forceinline__ unsigned wave_scan(unsigned v) {
    v += (unsigned)__builtin_amdgcn_update_dpp(0, (int)v, 0x111, 0xf, 0xf, false);
    v += (unsigned)__builtin_amdgcn_update_dpp(0, (int)v, 0x112, 0xf, 0xf, false);
    v += (unsigned)__builtin_amdgcn_update_dpp(0, (int)v, 0x114, 0xf, 0xf, false);
    v += (unsigned)__builtin_amdgcn_update_dpp(0, (int)v, 0x118, 0xf, 0xf, false);
    v += (unsigned)__builtin_amdgcn_update_dpp(0, (int)v, 0x142, 0xa, 0xf, false);
    v += (unsigned)__builtin_amdgcn_update_dpp(0, (int)v, 0x143, 0xc, 0xf, false);
    return v;
}

__device__ __forceinline__ int rot_of(int d) { return d == 0 ? 0 : ((d - 1) * 16) / 7; }

__device__ __forceinline__ bool give_up(unsigned long long t0, unsigned char* ws_own, unsigned call, unsigned& spins) {
    if ((++spins & 15u) != 0u) return false;
    if (ld_agent32(ws_own + C_TMO) == call || now() - t0 > DEADLINE) { st_agent32(ws_own + C_TMO, call); return true; }
    return false;
}

constexpr int SEND_CUS = 7;
constexpr int WL_T = 9216, WL_NAR = WL_T + TAIL_CAP + 16, WL_WID = WL_NAR + NAR_CAP * 2;
constexpr int WAVE_LDS = WL_WID + WID_CAP * 4;
typedef unsigned short u16x2 __attribute__((ext_vector_type(2)));
__device__ __forceinline__ u16x2 pk(unsigned v) { return __builtin_bit_cast(u16x2, v); }
__device__ __forceinline__ unsigned up(u16x2 v) { return __builtin_bit_cast(unsigned, v); }
template <int OFF> __device__ __forceinline__ void st_wt16o(void* p, u32x4 v) {
    asm volatile("global_store_dwordx4 %0, %1, off offset:%2 sc0 sc1\n\ts_nop 1" :: "v"(p), "v"(v), "i"(OFF) : "memory");
}
__device__ __forceinline__ void add_sys32(void* p) { (void)__hip_atomic_fetch_add((gu32*)p, 1u, __ATOMIC_RELAXED, __HIP_MEMORY_SCOPE_SYSTEM); }

struct PieceId { int isB, s, idx, pc; };
__device__ __forceinline__ PieceId piece_of(int q, int rot) {
    const int c = q >> 2, s = c / 24, u = c - 24 * s;
    PieceId id; id.isB = u < 2 * NCB && !(u & 1); id.s = s;
    id.idx = id.isB ? (((u >> 1) + (rot >> 1)) & 7) : (((u < 2 * NCB ? u >> 1 : u - NCB) + rot) & 15);
    id.pc = q & 3; return id;
}
__device__ __forceinline__ unsigned char* image_of(unsigned char* ws, int src, const PieceId& id, int lane, int& stride4) {
    stride4 = id.isB ? 16384 : 4096;
    return id.isB ? ws + OFF_B + (size_t)((src * NSL + id.s) * NCB + id.idx) * CHUNK + id.pc * 4096 + lane * 16
                  : ws + OFF_A + (size_t)((src * NSL + id.s) * NRB + id.idx) * CHUNK + id.pc * 8192 + lane * 16;
}
__device__ __forceinline__ unsigned char* counter_of(unsigned char* ws, int src, const PieceId& id) {
    return (id.isB ? ws + C_FB : ws + C_FA) + (size_t)(id.idx * NR + src) * 128 + id.s * 4;
}
__device__ __forceinline__ void store_image(unsigned char* img, int stride4, const u32x4 (&v)[8]) {
    st_wt16o<0>(img, v[0]); st_wt16o<1024>(img, v[1]); st_wt16o<2048>(img, v[2]); st_wt16o<3072>(img, v[3]);
    unsigned char* i2 = img + stride4;
    st_wt16o<0>(i2, v[4]); st_wt16o<1024>(i2, v[5]); st_wt16o<2048>(i2, v[6]); st_wt16o<3072>(i2, v[7]);
}
__device__ __forceinline__ void issue_piece(const Params& P, int dest, const PieceId& id, int lane, u32x4 (&r)[8]) {
    if (id.isB) {
        const unsigned char* src = (const unsigned char*)P.w + (size_t)(64 * id.s + (lane >> 3)) * (NN * 2) + (size_t)(NL * dest + 256 * id.idx + 64 * id.pc + (lane & 7) * 8) * 2;
#pragma unroll
        for (int it = 0; it < 8; ++it) r[it] = *(const GAS u32x4*)(src + (size_t)(it * 8) * (NN * 2));
    } else {
        const unsigned char* src = (const unsigned char*)P.x + (size_t)(256 * id.idx + 64 * id.pc + (lane & 15)) * (KL * 2) + (64 * id.s + 8 * (lane >> 4)) * 2;
#pragma unroll
        for (int ff = 0; ff < 8; ++ff) r[ff] = *(const GAS u32x4*)(src + (size_t)(16 * (ff >> 1)) * (KL * 2) + (ff & 1) * 64);
    }
}
__device__ __forceinline__ void finish_piece(const PieceId& id, int lane, LAS unsigned char* wl, u32x4 (&v)[8]) {
    if (id.isB) {
#pragma unroll
        for (int it = 0; it < 8; ++it) *(LAS u32x4*)(wl + (it * 8 + (lane >> 3)) * 144 + (lane & 7) * 16) = v[it];
        WAIT_L(0);
        const int fg = lane >> 4, q4 = (lane >> 2) & 3, p4 = lane & 3;
        LAS unsigned char* a0 = wl + (8 * fg + q4) * 144 + p4 * 32;
#pragma unroll
        for (int ff = 0; ff < 8; ++ff) {
            const int hn = ff >> 1, kf = ff & 1;
            LAS unsigned char* a = a0 + (32 * kf) * 144 + hn * 8;
            typedef short v4i16_t __attribute__((ext_vector_type(4)));
            const v4i16_t lo = __builtin_amdgcn_ds_read_tr16_b64_v4i16((LAS v4i16_t*)a);
            const v4i16_t hi = __builtin_amdgcn_ds_read_tr16_b64_v4i16((LAS v4i16_t*)(a + 4 * 144));
            const u32x2 l2 = __builtin_bit_cast(u32x2, lo), h2 = __builtin_bit_cast(u32x2, hi);
            v[ff] = (u32x4){l2.x, l2.y, h2.x, h2.y};
        }
        WAIT_L(0);
    }
}

constexpr unsigned META_RAW = 0x100u;
__device__ __forceinline__ bool pack_piece(const u32x4 (&v)[8], int lane, LAS unsigned char* wl, unsigned char* sl, unsigned& meta) {
    unsigned char* a = sl + lane * 16;
#pragma unroll
    for (int g = 0; g < 4; ++g) {
        u32x4 sm;
#pragma unroll
        for (int h = 0; h < 2; ++h) {
            unsigned t[4];
#pragma unroll
            for (int i = 0; i < 4; ++i) { const unsigned dw = v[2 * g + h][i]; t[i] = (dw & 0x007f007fu) | ((dw >> 8) & ~0x007f007fu); }
            sm[2 * h] = __builtin_amdgcn_perm(t[1], t[0], 0x06040200u);
            sm[2 * h + 1] = __builtin_amdgcn_perm(t[3], t[2], 0x06040200u);
        }
        if (g == 0) st_wt16o<0>(a, sm); else if (g == 1) st_wt16o<1024>(a, sm); else if (g == 2) st_wt16o<2048>(a, sm); else st_wt16o<3072>(a, sm);
    }
    u16x2 mx = pk(0u);
#pragma unroll
    for (int ff = 0; ff < 8; ++ff) {
        *(LAS u32x4*)(wl + ff * 1024 + lane * 16) = v[ff];
#pragma unroll
        for (int i = 0; i < 4; ++i) mx = __builtin_elementwise_max(mx, pk(v[ff][i] & 0x7fff7fffu));
    }
    *(LAS u32x4*)(wl + WL_T + lane * 16) = (u32x4){0u, 0u, 0u, 0u};
    *(LAS u32x4*)(wl + WL_T + 1024 + lane * 16) = (u32x4){0u, 0u, 0u, 0u};
    const unsigned lmax = max((unsigned)mx.x, (unsigned)mx.y) >> 7, hi = wave_umax(lmax);
    const unsigned top = (hi > 0u && __builtin_popcountll(__ballot(lmax == hi)) <= 2) ? hi - 1u : hi;
    const unsigned top2 = top * 0x00010001u;
    u32x4 pp; unsigned f[8], len[8], mk[2] = {0u, 0u};
    unsigned c1 = 0x00010001u, c3 = 0x00030003u, c7 = 0x00070007u, c10 = 0x000a000au, k3 = 3u;
    asm volatile("" : "+s"(c1), "+s"(c3), "+s"(c7), "+s"(c10), "+s"(k3));
#pragma unroll
    for (int ff = 0; ff < 8; ++ff) {
        unsigned pw = 0, fw = 0, pos = 0;
#pragma unroll
        for (int i = 0; i < 4; ++i) {
            const u16x2 r = pk(top2) - pk((v[ff][i] >> 7) & 0x00ff00ffu);
            const unsigned p2 = up(__builtin_elementwise_min(r - pk(c1), pk(c3)));
            const unsigned s2 = up(__builtin_elementwise_min(__builtin_elementwise_sub_sat(r, pk(c3)), pk(c7)));
            const unsigned e3 = __umul24(p2 & (p2 >> 1), k3);
            pw |= p2 << (2 * i);
            mk[ff >> 2] |= up(__builtin_elementwise_min(__builtin_elementwise_sub_sat(r, pk(c10)), pk(c1))) << (4 * (ff & 3) + i);
            const unsigned lo = (s2 & 0xffffu) << pos; pos += e3 & 0xffffu;
            const unsigned hi2 = (s2 >> 16) << pos; pos += e3 >> 16;
            fw |= lo | hi2;
        }
        if (ff & 1) pp[ff >> 1] |= pw << 8; else pp[ff >> 1] = pw;
        f[ff] = fw; len[ff] = pos;
    }
    st_wt16o<0>(a + PK_P, pp);
    const unsigned ltot = ((len[0] + len[1]) + (len[2] + len[3])) + ((len[4] + len[5]) + (len[6] + len[7]));
    const unsigned incl = wave_scan(ltot), sbits = __builtin_amdgcn_readlane((int)incl, 63);
    {
        unsigned o = incl - ltot;
#pragma unroll
        for (int ff = 0; ff < 8; ++ff) {
            const unsigned long long x = (unsigned long long)f[ff] << (o & 31u);
            LAS unsigned* d = (LAS unsigned*)(wl + WL_T + (o >> 5) * 4u);
            (void)__hip_atomic_fetch_or(d, (unsigned)x, __ATOMIC_RELAXED, __HIP_MEMORY_SCOPE_WORKGROUP);
            (void)__hip_atomic_fetch_or(d + 1, (unsigned)(x >> 32), __ATOMIC_RELAXED, __HIP_MEMORY_SCOPE_WORKGROUP);
            o += len[ff];
        }
    }
    const unsigned nbase = ((sbits + 31u) >> 5) * 4u;
    LAS unsigned short* nar = (LAS unsigned short*)(wl + WL_T + nbase);
    LAS unsigned* wid = (LAS unsigned*)(wl + WL_WID);
    unsigned nn = 0, nw = 0;
    while (__ballot((mk[0] | mk[1]) != 0u)) {
        const bool act = (mk[0] | mk[1]) != 0u, g = mk[0] == 0u;
        const unsigned cur = g ? mk[1] : mk[0];
        const unsigned bit = act ? (unsigned)__builtin_ctz(cur) : 0u;
        const unsigned idx = (((g ? 4u : 0u) + ((bit >> 2) & 3u)) * 64u + (unsigned)lane) * 8u + 2u * (bit & 3u) + (bit >> 4);
        const unsigned val = *(LAS unsigned short*)(wl + idx * 2u);
        const unsigned d = top - ((val >> 7) & 0xffu) - 10u;
        const bool wide = act && d > 15u;
        const unsigned long long bn = __ballot(act && !wide), bw = __ballot(wide);
        const unsigned pos = nn + __builtin_amdgcn_mbcnt_hi((unsigned)(bn >> 32), __builtin_amdgcn_mbcnt_lo((unsigned)bn, 0u));
        if (act && !wide && pos < (unsigned)NAR_CAP) nar[pos] = (unsigned short)((idx << 4) | d);
        nn += (unsigned)__builtin_popcountll(bn);
        if (bw) {
            const unsigned pw = nw + __builtin_amdgcn_mbcnt_hi((unsigned)(bw >> 32), __builtin_amdgcn_mbcnt_lo((unsigned)bw, 0u));
            if (wide && pw < (unsigned)WID_CAP) wid[pw] = (idx << 16) | val;
            nw += (unsigned)__builtin_popcountll(bw);
        }
        if (g) mk[1] = cur & (cur - 1u); else mk[0] = cur & (cur - 1u);
        if (nn > (unsigned)NAR_CAP || nw > (unsigned)WID_CAP) return false;
    }
    WAIT_L(0);
    const unsigned tlen = nbase + 2u * nn;
    if (16u * lane < tlen) st_wt16o<0>(a + PK_T, *(volatile LAS u32x4*)(wl + WL_T + lane * 16));
    if (tlen > 1024u && 1024u + 16u * lane < tlen) st_wt16o<1024>(a + PK_T, *(volatile LAS u32x4*)(wl + WL_T + 1024 + lane * 16));
    if (4u * lane < nw) st_wt16o<0>(a + PK_WID, *(volatile LAS u32x4*)(wl + WL_WID + lane * 16));
    meta = top | (nn << 9) | (nw << 18);
    return true;
}
__device__ __forceinline__ void unpack_piece(const unsigned char* sl, unsigned meta, int lane, LAS unsigned char* wl, u32x4 (&v)[8]) {
    u32x4 k0, k1, k2, k3, pp, t0, t1; unsigned wd;
    const unsigned char* a = sl + lane * 16;
    asm volatile("global_load_dwordx4 %4, %9, off sc0 sc1\n\t"
                 "global_load_dwordx4 %5, %9, off offset:1024 sc0 sc1\n\t"
                 "global_load_dwordx4 %6, %9, off offset:2048 sc0 sc1\n\t"
                 "global_load_dwordx4 %0, %8, off sc0 sc1\n\t"
                 "global_load_dwordx4 %1, %8, off offset:1024 sc0 sc1\n\t"
                 "global_load_dwordx4 %2, %8, off offset:2048 sc0 sc1\n\t"
                 "global_load_dwordx4 %3, %8, off offset:3072 sc0 sc1\n\t"
                 "global_load_dword %7, %10, off sc0 sc1\n\t"
                 "s_waitcnt vmcnt(0)"
                 : "=&v"(k0), "=&v"(k1), "=&v"(k2), "=&v"(k3), "=&v"(pp), "=&v"(t0), "=&v"(t1), "=&v"(wd)
                 : "v"(a), "v"(a + PK_P), "v"(sl + PK_WID + lane * 4) : "memory");
    static_assert(PK_T == PK_P + 1024, "tail follows the code plane");
    *(LAS u32x4*)(wl + WL_T + lane * 16) = t0;
    *(LAS u32x4*)(wl + WL_T + 1024 + lane * 16) = t1;
    const u32x4 sm[4] = {k0, k1, k2, k3};
    unsigned es[4], len[8];
#pragma unroll
    for (int g = 0; g < 4; ++g) {
        es[g] = pp[g] & (pp[g] >> 1) & 0x55555555u;
        len[2 * g] = 3u * (unsigned)__builtin_popcount(es[g] & 0x00ff00ffu);
        len[2 * g + 1] = 3u * (unsigned)__builtin_popcount(es[g] & 0xff00ff00u);
    }
    const unsigned ltot = ((len[0] + len[1]) + (len[2] + len[3])) + ((len[4] + len[5]) + (len[6] + len[7]));
    const unsigned incl = wave_scan(ltot), sbits = __builtin_amdgcn_readlane((int)incl, 63);
    unsigned f[8];
    {
        unsigned o = incl - ltot;
#pragma unroll
        for (int ff = 0; ff < 8; ++ff) {
            const LAS unsigned* d = (const LAS unsigned*)(wl + WL_T + (o >> 5) * 4u);
            f[ff] = __builtin_amdgcn_alignbit(d[1], d[0], o & 31u);
            o += len[ff];
        }
    }
    const unsigned top = meta & 0xffu, tb = top * 0x01010101u;
    const unsigned te0 = up(pk(tb & 0x00ff00ffu) - pk(0x00050000u)) & 0x00ff00ffu | ((up(pk((tb >> 8) & 0x00ff00ffu) - pk(0x00060004u)) & 0x00ff00ffu) << 8);
    const unsigned te1 = up(pk(tb & 0x00ff00ffu) - pk(0x00090007u)) & 0x00ff00ffu | ((up(pk((tb >> 8) & 0x00ff00ffu) - pk(0x000a0008u)) & 0x00ff00ffu) << 8);
    const unsigned tn = up(pk(tb & 0x00ff00ffu) - pk(0x00030001u)) & 0x00ff00ffu | ((up(pk((tb >> 8) & 0x00ff00ffu) - pk(0x00000002u)) & 0x00ff00ffu) << 8);
    unsigned three = 3u, kff = 0xffu;
    asm volatile("" : "+s"(three), "+s"(kff));
#pragma unroll
    for (int ff = 0; ff < 8; ++ff) {
        u32x4 o; unsigned pos = 0;
        const unsigned pw = pp[ff >> 1] >> (8 * (ff & 1)), ew = es[ff >> 1] >> (8 * (ff & 1));
#pragma unroll
        for (int i = 0; i < 4; ++i) {
            const unsigned p2 = (pw >> (2 * i)) & 0x00030003u, e2 = (ew >> (2 * i)) & 0x00010001u, e3 = __umul24(e2, three);
            const unsigned slo = __builtin_amdgcn_ubfe(f[ff], pos, 3u); pos += e3 & 0xffffu;
            const unsigned shi = __builtin_amdgcn_ubfe(f[ff], pos, 3u); pos += e3 >> 16;
            const unsigned xe = __builtin_amdgcn_perm(te1, te0, slo | (shi << 16));
            const unsigned xn = __builtin_amdgcn_perm(tn, tn, p2);
            const unsigned m = __umul24(e2, kff);
            const unsigned ex = ((xe & m) | (xn & ~m)) & 0x00ff00ffu;
            const unsigned s4 = sm[ff >> 1][(ff & 1) * 2 + (i >> 1)];
            const unsigned u = __builtin_amdgcn_perm(s4, s4, (i & 1) ? 0x03030202u : 0x01010000u);
            o[i] = (ex << 7) | (u & 0x807f807fu);
        }
        *(LAS u32x4*)(wl + ff * 1024 + lane * 16) = o;
    }
    const unsigned nn = (meta >> 9) & 0x1ffu, nw = meta >> 18;
    const LAS unsigned short* nar = (const LAS unsigned short*)(wl + WL_T + ((sbits + 31u) >> 5) * 4u);
    for (unsigned e = (unsigned)lane; e < nn; e += 64u) {
        const unsigned ent = nar[e], idx = ent >> 4;
        (void)__hip_atomic_fetch_sub((LAS unsigned*)(wl + (idx >> 1) * 4), ((ent & 15u) << 7) << (16 * (idx & 1u)), __ATOMIC_RELAXED, __HIP_MEMORY_SCOPE_WORKGROUP);
    }
    if ((unsigned)lane < nw) *(LAS unsigned short*)(wl + ((wd >> 16) & 4095u) * 2) = (unsigned short)wd;
    WAIT_L(0);
#pragma unroll
    for (int ff = 0; ff < 8; ++ff) v[ff] = *(LAS u32x4*)(wl + ff * 1024 + lane * 16);
}

constexpr int NUCU = 16 - SEND_CUS;
constexpr int NUW = NUCU * 8 + 18;
constexpr int NSELF = 128 + 7 * NUCU * 8;

__device__ __forceinline__ void comm(const Params& P, int b, unsigned callv, LAS unsigned char* lds) {
    const int tid = threadIdx.x, wid = __builtin_amdgcn_readfirstlane(tid >> 6), lane = tid & 63;
    const int rank = P.rank;
    const int xcd = b & 7, slot = b >> 3;
    unsigned char* own = P.ws[rank];
    LAS unsigned char* wl = lds + wid * WAVE_LDS;
    if (xcd != 0 && slot < SEND_CUS) {
        const int d = xcd, dest = (rank + d) & 7, rot = rot_of(d);
        constexpr int nw = SEND_CUS * 8;
        unsigned char* wsd = P.ws[dest];
        gu32* tick = (gu32*)(own + C_TICK + (size_t)d * 128);
        u32x4 nxt[8];
        int q = slot * 8 + wid;
        issue_piece(P, dest, piece_of(q, rot), lane, nxt);
        const unsigned call = __builtin_amdgcn_readfirstlane(callv);
        const unsigned tbase = (call - 1u) * (unsigned)NPC - (unsigned)nw;
        unsigned tk = 0;
        if (lane == 0) tk = __hip_atomic_fetch_add(tick, 1u, __ATOMIC_RELAXED, __HIP_MEMORY_SCOPE_AGENT);
        unsigned char* pub = nullptr; unsigned long long pubv = 0;
        for (;;) {
            const PieceId id = piece_of(q, rot);
            u32x4 v[8];
            asm volatile("s_waitcnt vmcnt(0)" : "+v"(nxt[0]), "+v"(nxt[1]), "+v"(nxt[2]), "+v"(nxt[3]), "+v"(nxt[4]), "+v"(nxt[5]), "+v"(nxt[6]), "+v"(nxt[7]), "+v"(tk) :: "memory");
            if (pub && lane == 0) __hip_atomic_store((gu64*)pub, pubv, __ATOMIC_RELAXED, __HIP_MEMORY_SCOPE_SYSTEM);
#pragma unroll
            for (int ff = 0; ff < 8; ++ff) v[ff] = nxt[ff];
            const unsigned qn = (unsigned)__builtin_amdgcn_readfirstlane(tk) - tbase;
            const bool more = qn < (unsigned)NPC;
            if (more) {
                issue_piece(P, dest, piece_of((int)qn, rot), lane, nxt);
                if (lane == 0) tk = __hip_atomic_fetch_add(tick, 1u, __ATOMIC_RELAXED, __HIP_MEMORY_SCOPE_AGENT);
            }
            finish_piece(id, lane, wl, v);
            unsigned meta = META_RAW;
            unsigned char* sl = wsd + OFF_P + (size_t)(rank * NPC + q) * SLOT;
            if (!pack_piece(v, lane, wl, sl, meta)) { meta = META_RAW; int st4; unsigned char* img = image_of(wsd, rank, id, lane, st4); store_image(img, st4, v); }
            pub = sl + PK_GR;
            pubv = ((unsigned long long)call << 32) | meta;
            if (!more) break;
            q = (int)qn;
        }
        WAIT_V(0);
        if (lane == 0) __hip_atomic_store((gu64*)pub, pubv, __ATOMIC_RELAXED, __HIP_MEMORY_SCOPE_SYSTEM);
    } else {
        const int g0 = xcd == 0 ? slot * 8 + wid : 128 + ((xcd - 1) * NUCU + slot - SEND_CUS) * 8 + wid;
        for (int q = g0; q < NPC; q += NSELF) {
            const PieceId id = piece_of(q, 0);
            u32x4 v[8];
            issue_piece(P, rank, id, lane, v);
            finish_piece(id, lane, wl, v);
            int st4; unsigned char* img = image_of(own, rank, id, lane, st4);
            store_image(img, st4, v);
            WAIT_V(0);
            if (lane == 0) add_sys32(counter_of(own, rank, id));
        }
        const unsigned call = __builtin_amdgcn_readfirstlane(callv);
        const int d = xcd != 0 ? xcd : 1 + g0 % 7;
        const int first = xcd != 0 ? (slot - SEND_CUS) * 8 + wid : NUCU * 8 + g0 / 7;
        const int src = (rank - d) & 7, rot = rot_of(d);
        constexpr int NUP = NPC;
        gu32* tick = (gu32*)(own + C_TICK + (size_t)(NR + d) * 128);
        const unsigned tbase = (call - 1u) * (unsigned)NUP - (unsigned)NUW;
        unsigned tk = 0;
        for (int q = first < NUW ? first : NUP; q < NUP;) {
            if (lane == 0) tk = __hip_atomic_fetch_add(tick, 1u, __ATOMIC_RELAXED, __HIP_MEMORY_SCOPE_AGENT);
            const PieceId id = piece_of(q, rot);
            const unsigned char* sl = own + OFF_P + (size_t)(src * NPC + q) * SLOT;
            unsigned meta;
            {
                const unsigned long long t0 = now();
                unsigned spins = 0;
                for (;;) {
                    const unsigned long long g = __hip_atomic_load((gu64*)(sl + PK_GR), __ATOMIC_RELAXED, __HIP_MEMORY_SCOPE_SYSTEM);
                    const unsigned tag = __builtin_amdgcn_readfirstlane((unsigned)(g >> 32));
                    meta = __builtin_amdgcn_readfirstlane((unsigned)g);
                    if (tag >= call) break;
                    if (give_up(t0, own, call, spins)) { meta = META_RAW; break; }
                    __builtin_amdgcn_s_sleep(2);
                }
            }
            if (!(meta & META_RAW)) {
                u32x4 v[8];
                unpack_piece(sl, meta, lane, wl, v);
                int st4; unsigned char* img = image_of(own, src, id, lane, st4);
                store_image(img, st4, v);
                WAIT_V(0);
            }
            if (lane == 0) add_sys32(counter_of(own, src, id));
            const unsigned qn = (unsigned)__builtin_amdgcn_readfirstlane(tk) - tbase;
            q = qn < (unsigned)NUP ? (int)qn : NUP;
        }
    }
    if (b == 0) {
        const unsigned call = __builtin_amdgcn_readfirstlane(callv);
        if (wid < 2) {
            const unsigned char* f = own + C_DONE + (size_t)(tid >> 4) * 128 + (tid & 15) * 4;
            const unsigned long long t0 = now();
            bool bad = false; unsigned spins = 0;
            while (!__all(ld_sys32(f) >= call)) {
                if (give_up(t0, own, call, spins)) { bad = true; break; }
                __builtin_amdgcn_s_sleep(2);
            }
            if (bad && lane == 0) P.out[rank][rank * NL] = (signed char)-128;
        }
    }
}

__device__ __forceinline__ void gemm(const Params& P, int b, unsigned call, LAS unsigned char* lds) {
    const int tid = threadIdx.x, wid = __builtin_amdgcn_readfirstlane(tid >> 6), lane = tid & 63, wr = wid >> 2, wc = wid & 3;
    const int rank = P.rank;
    const int xcd = b & 7, slot = b >> 3;
    const int ti = 2 * ((rank + xcd) & 7) + (slot >> 3), tj = slot & 7;
    unsigned char* own = P.ws[rank];
    const unsigned char* abase = uni(own + OFF_A + (size_t)ti * CHUNK);
    const unsigned char* bbase = uni(own + OFF_B + (size_t)tj * CHUNK);
    auto aptr = [&](int k) -> const unsigned char* { return abase + ((size_t)(unsigned)k << 19); };
    auto bptr = [&](int k) -> const unsigned char* { return bbase + ((size_t)(unsigned)k << 18); };
    static_assert(NRB * CHUNK == (1 << 19) && NCB * CHUNK == (1 << 18), "chunk strides");

    LAS unsigned* ctl = (LAS unsigned*)(lds + LDS_CTL);
    const unsigned ctl_addr = (unsigned)(size_t)ctl;
    const unsigned char* fbaseA = uni(own + C_FA + (size_t)ti * NR * 128);
    const unsigned char* fbaseB = uni(own + C_FB + (size_t)tj * NR * 128);
    const unsigned foff = (unsigned)(lane >> 3) * 128u + (unsigned)(lane & 7) * 4u;
    const unsigned target = 4u * call;
    unsigned long long known = 0, taken = 0;
    auto pick = [&]() -> int {
        if ((known & ~taken) == 0) {
            if (taken == ~0ull) return 0;
            if (wid == 0) {
                const unsigned long long t0 = now();
                unsigned long long m; unsigned spins = 0;
                for (;;) {
                    unsigned va, vb;
                    asm volatile("global_load_dword %0, %2, %3 sc0 sc1\n\tglobal_load_dword %1, %2, %4 sc0 sc1\n\ts_waitcnt vmcnt(0)"
                                 : "=&v"(va), "=&v"(vb) : "v"(foff), "s"(fbaseA), "s"(fbaseB) : "memory");
                    m = __ballot(va >= target && vb >= target);
                    if (m & ~taken) break;
                    if (give_up(t0, own, call, spins)) { m = ~0ull; break; }
                    __builtin_amdgcn_s_sleep(1);
                }
                const u32x2 mv = {(unsigned)m, (unsigned)(m >> 32)};
                asm volatile("ds_write_b64 %0, %1\n\ts_waitcnt lgkmcnt(0)" :: "v"(ctl_addr), "v"(mv) : "memory");
            }
            CFENCE; BAR; CFENCE;
            u32x2 r;
            asm volatile("ds_read_b64 %0, %1\n\ts_waitcnt lgkmcnt(0)" : "=v"(r) : "v"(ctl_addr) : "memory");
            known |= ((unsigned long long)(unsigned)__builtin_amdgcn_readfirstlane(r.y) << 32) | (unsigned)__builtin_amdgcn_readfirstlane(r.x);
        }
        const int k = __builtin_ctzll(known & ~taken);
        taken |= 1ull << k;
        return k;
    };
    unsigned char* amax_dst = P.ws[lane & 7] + C_AMAX + (size_t)(rank * NGEMM + b) * 8;
    asm volatile("" : "+v"(amax_dst));
    const int dest = ti >> 1;
    signed char* out_tile = uni(P.out[dest] + (size_t)((ti & 1) * 256) * NN + (size_t)rank * NL + tj * 256);
    unsigned char* done_flag = uni(P.ws[dest] + C_DONE + (size_t)rank * 128 + ((ti & 1) * 8 + tj) * 4);

    f32x4 acc[2][2][4][2];
#pragma unroll
    for (int a = 0; a < 2; ++a)
#pragma unroll
        for (int c = 0; c < 2; ++c)
#pragma unroll
            for (int m = 0; m < 4; ++m)
#pragma unroll
                for (int n = 0; n < 2; ++n) acc[a][c][m][n] = (f32x4){0.f, 0.f, 0.f, 0.f};
    bf16x8 At[4][2], B0[2][2], B1[2][2];
    const unsigned voff = (unsigned)wid * 2048u + (unsigned)lane * 16u;
    const unsigned ldsw = (unsigned)(size_t)lds + (unsigned)wid * 2048u;
    const int aoff = wr * 8192 + lane * 16, boff = wc * 4096 + lane * 16;
    constexpr int HTB = 16384;
#define SA(b_, h_) (((b_) * 2 + (h_)) * HTB)
#define SB(b_, h_) ((4 + (b_) * 2 + (h_)) * HTB)
#define STAGE(bufoff, gbase) asm volatile("s_mov_b32 m0, %2\n\ts_nop 0\n\tglobal_load_lds_dwordx4 %0, %1 sc0 sc1\n\tglobal_load_lds_dwordx4 %0, %1 offset:1024 sc0 sc1" \
        :: "v"(voff), "s"(gbase), "s"(ldsw + (bufoff)) : "memory")
#define LDA(dst, b_, h_) do { _Pragma("unroll") for (int m = 0; m < 4; ++m) _Pragma("unroll") for (int k = 0; k < 2; ++k) dst[m][k] = *(const LAS bf16x8*)(lds + SA(b_, h_) + aoff + m * 2048 + k * 1024); } while (0)
#define LDB(dst, b_, h_) do { _Pragma("unroll") for (int n = 0; n < 2; ++n) _Pragma("unroll") for (int k = 0; k < 2; ++k) dst[n][k] = *(const LAS bf16x8*)(lds + SB(b_, h_) + boff + n * 2048 + k * 1024); } while (0)
#define MMA(ai, bj, At_, Bt_) do { __builtin_amdgcn_s_setprio(1); _Pragma("unroll") for (int m = 0; m < 4; ++m) _Pragma("unroll") for (int n = 0; n < 2; ++n) _Pragma("unroll") for (int k = 0; k < 2; ++k) \
        acc[ai][bj][m][n] = __builtin_amdgcn_mfma_f32_16x16x32_bf16(Bt_[n][k], At_[m][k], acc[ai][bj][m][n], 0, 0, 0); __builtin_amdgcn_s_setprio(0); } while (0)

    {
        const int k0 = pick();
        const unsigned char* a0 = aptr(k0); const unsigned char* b0 = bptr(k0);
        STAGE(SB(0, 0), b0); STAGE(SB(0, 1), b0 + HTB); STAGE(SA(0, 0), a0); STAGE(SA(0, 1), a0 + HTB);
    }
    int k1 = pick();
    if (wr == 1) BAR;
    WAIT_V(2); BAR;
    {
        const unsigned char* a1 = aptr(k1); const unsigned char* b1 = bptr(k1);
        STAGE(SB(1, 0), b1); STAGE(SA(1, 0), a1); STAGE(SB(1, 1), b1 + HTB);
    }
    WAIT_V(6); BAR;
#define LOOK_ISSUE() do { if (look) asm volatile("global_load_dword %0, %2, %3 sc0 sc1\n\tglobal_load_dword %1, %2, %4 sc0 sc1" : "=&v"(pa), "=&v"(pb) : "v"(foff), "s"(fbaseA), "s"(fbaseB) : "memory"); } while (0)
#define LOOK_PUBLISH(OFF) do { if (wid == 0) { unsigned long long m_ = known; \
            if (look) { asm volatile("" : "+v"(pa), "+v"(pb)); m_ |= __ballot(pa >= target && pb >= target); } \
            const u32x2 mv_ = {(unsigned)m_, (unsigned)(m_ >> 32)}; \
            asm volatile("ds_write_b64 %0, %1 offset:" #OFF :: "v"(ctl_addr), "v"(mv_) : "memory"); } } while (0)
#define LOOK_READ(OFF) asm volatile("ds_read_b64 %0, %1 offset:" #OFF : "=v"(rn) : "v"(ctl_addr) : "memory")
#define LOOK_TAKE() do { asm volatile("" : "+v"(rn)); known |= ((unsigned long long)(unsigned)__builtin_amdgcn_readfirstlane(rn.y) << 32) | (unsigned)__builtin_amdgcn_readfirstlane(rn.x); } while (0)
    for (int t = 0; t < NKT; t += 2) {
        const unsigned char* a1 = aptr(k1);
        unsigned pa = 0, pb = 0;
        const bool look = wid == 0 && known != ~0ull;
        u32x2 rn;
        LOOK_ISSUE();
        LDB(B0, 0, 0); LDB(B1, 0, 1); SCHED; LDA(At, 0, 0); STAGE(SA(1, 1), a1 + HTB);
        WAIT_V(8); WAIT_L(0); BAR; MMA(0, 0, At, B0); MMA(0, 1, At, B1); BAR; SCHED;
        const int k2 = pick();
        const unsigned char* a2 = aptr(k2); const unsigned char* b2 = bptr(k2);
        LDA(At, 0, 1); STAGE(SB(0, 0), b2); STAGE(SB(0, 1), b2 + HTB); STAGE(SA(0, 0), a2);
        WAIT_V(8); LOOK_PUBLISH(8);
        WAIT_L(0); BAR; MMA(1, 0, At, B0); MMA(1, 1, At, B1); BAR; SCHED;
        LOOK_READ(8);
        LDB(B0, 1, 0); LDB(B1, 1, 1); SCHED; LDA(At, 1, 0); STAGE(SA(0, 1), a2 + HTB);
        WAIT_V(8); WAIT_L(0); LOOK_TAKE();
        BAR; MMA(0, 0, At, B0); MMA(0, 1, At, B1); BAR; SCHED;
        const int k3 = pick();
        const unsigned char* a3 = aptr(k3); const unsigned char* b3 = bptr(k3);
        LDA(At, 1, 1); STAGE(SB(1, 0), b3); STAGE(SB(1, 1), b3 + HTB); STAGE(SA(1, 0), a3);
        WAIT_V(8); WAIT_L(0); BAR; MMA(1, 0, At, B0); MMA(1, 1, At, B1); BAR; SCHED;
        k1 = k3;
    }
#undef LOOK_ISSUE
#undef LOOK_PUBLISH
#undef LOOK_READ
#undef LOOK_TAKE
    if (wr == 0) BAR;

    int tz = threadIdx.x; asm volatile("" : "+v"(tz));
    const int lane2 = tz & 63, fr = lane2 & 15, fq = lane2 >> 4;
    float am;
    asm volatile("s_nop 15\n\tv_mov_b32 %0, 0" : "=v"(am));
#pragma unroll
    for (int a = 0; a < 2; ++a)
#pragma unroll
        for (int c = 0; c < 2; ++c)
#pragma unroll
            for (int m = 0; m < 4; ++m)
#pragma unroll
                for (int n = 0; n < 2; ++n) {
                    const f32x4 v = acc[a][c][m][n];
                    asm("v_max3_f32 %0, |%1|, |%2|, %0" : "+v"(am) : "v"(v[0]), "v"(v[1]));
                    asm("v_max3_f32 %0, |%1|, |%2|, %0" : "+v"(am) : "v"(v[2]), "v"(v[3]));
                }
    LAS unsigned* red = (LAS unsigned*)(lds + LDS_CTL + 64);
    { const unsigned wm = wave_umax(__float_as_uint(am)); if (lane2 == 0) red[wid] = wm; }
    WAIT_L(0); BAR; CFENCE;
    if (tz < NR) {
        const u32x4 r0 = *(LAS u32x4*)red, r1 = *(LAS u32x4*)(red + 4);
        const unsigned m8 = max(max(max(r0.x, r0.y), max(r0.z, r0.w)), max(max(r1.x, r1.y), max(r1.z, r1.w)));
        __hip_atomic_store((gu64*)amax_dst, ((unsigned long long)call << 32) | m8, __ATOMIC_RELAXED, __HIP_MEMORY_SCOPE_SYSTEM);
    }
    float gm; bool poison;
    {
        gu64* g0 = (gu64*)(own + C_AMAX) + tz;
        const unsigned long long t0 = now();
        unsigned long long a0, a1; unsigned tmo;
        for (;;) {
            a0 = __hip_atomic_load(g0, __ATOMIC_RELAXED, __HIP_MEMORY_SCOPE_SYSTEM);
            a1 = __hip_atomic_load(g0 + 512, __ATOMIC_RELAXED, __HIP_MEMORY_SCOPE_SYSTEM);
            tmo = ld_agent32(own + C_TMO);
            if (__all((unsigned)(a0 >> 32) >= call && (unsigned)(a1 >> 32) >= call)) break;
            if (tmo == call || now() - t0 > DEADLINE) { st_agent32(own + C_TMO, call); tmo = call; break; }
        }
        const unsigned wm = wave_umax(max((unsigned)a0, (unsigned)a1));
        if (lane2 == 0) { red[8 + wid] = wm; red[16 + wid] = tmo == call; }
        WAIT_L(0); BAR; CFENCE;
        const u32x4 r0 = *(LAS u32x4*)(red + 8), r1 = *(LAS u32x4*)(red + 12), p0 = *(LAS u32x4*)(red + 16), p1 = *(LAS u32x4*)(red + 20);
        gm = __uint_as_float(max(max(max(r0.x, r0.y), max(r0.z, r0.w)), max(max(r1.x, r1.y), max(r1.z, r1.w))));
        poison = ((p0.x | p0.y | p0.z | p0.w) | (p1.x | p1.y | p1.z | p1.w)) != 0u;
    }
    const float inv = gm > 0.f ? 127.0f / gm : 0.f;
    constexpr float MAGIC = 12582912.0f;
    signed char* obase = out_tile + (size_t)(wr * 64 + fr) * NN + wc * 64 + fq * 16;
#pragma unroll
    for (int a = 0; a < 2; ++a)
#pragma unroll
        for (int m = 0; m < 4; ++m) {
            u32x4 o;
#pragma unroll
            for (int c = 0; c < 2; ++c)
#pragma unroll
                for (int n = 0; n < 2; ++n) {
                    const f32x4 v = acc[a][c][m][n];
                    unsigned q[4];
#pragma unroll
                    for (int e = 0; e < 4; ++e)
                        q[e] = __float_as_uint(__builtin_amdgcn_fmed3f(__builtin_fmaf(v[e], inv, MAGIC), MAGIC - 127.0f, MAGIC + 127.0f));
                    const unsigned lo = __builtin_amdgcn_perm(q[1], q[0], 0x0c0c0400u), hi = __builtin_amdgcn_perm(q[3], q[2], 0x04000c0cu);
                    o[c * 2 + n] = lo | hi;
                }
            if (poison) o = (u32x4){0x80808080u, 0x80808080u, 0x80808080u, 0x80808080u};
            st_wt16(obase + (size_t)(a * 128 + m * 16) * NN, o);
        }
    WAIT_V(0);
    __syncthreads();
    if (tz == 0) st_sys32(done_flag, call);
#undef SA
#undef SB
#undef STAGE
#undef LDA
#undef LDB
#undef MMA
}

__global__ __launch_bounds__(NTHR, 2) void gemm_rs_q8_kernel(Params P) {
    extern __shared__ __attribute__((aligned(16))) unsigned char lds_raw[];
    LAS unsigned char* lds = (LAS unsigned char*)lds_raw;
    unsigned callv = 0;
    if ((threadIdx.x & 63) == 0)
        callv = __hip_atomic_fetch_add((gu32*)(P.ws[P.rank] + C_CNT + (size_t)(blockIdx.x * 8 + (threadIdx.x >> 6)) * 16), 1u, __ATOMIC_RELAXED, __HIP_MEMORY_SCOPE_AGENT) + 1u;
    if (blockIdx.x < NSEND) comm(P, blockIdx.x, callv, lds);
    else gemm(P, blockIdx.x - NSEND, (unsigned)__builtin_amdgcn_readfirstlane(callv), lds);
}

}

extern "C" void kernel_launch(int rank, int nranks, ncclComm_t comm,
                              hipStream_t stream,
                              void* const* d_in, const int* in_sizes, int n_in,
                              void* d_out, int out_size,
                              const int* in_dtypes, int out_dtype,
                              void* const* d_out_all,
                              void* const* d_ws_all, size_t ws_bytes) {
    (void)comm; (void)in_dtypes; (void)out_dtype; (void)d_out;
    if (nranks != NR || n_in != 2 || rank < 0 || rank >= NR || in_sizes[0] != MM * KL || in_sizes[1] != KL * NN ||
        out_size != ML * NN || ws_bytes < WS_USED) {
        fprintf(stderr, "kernel_launch: unexpected problem shape\n");
        return;
    }
    Params P;
    P.x = (const unsigned short*)d_in[0];
    P.w = (const unsigned short*)d_in[1];
    for (int p = 0; p < NR; ++p) { P.ws[p] = (unsigned char*)d_ws_all[p]; P.out[p] = (signed char*)d_out_all[p]; }
    P.rank = rank; P.pad = 0;
    HIP_CHECK(hipFuncSetAttribute((const void*)gemm_rs_q8_kernel, hipFuncAttributeMaxDynamicSharedMemorySize, LDS_BYTES));
    hipLaunchKernelGGL(gemm_rs_q8_kernel, dim3(NBLK), dim3(NTHR), LDS_BYTES, stream, P);
    HIP_CHECK(hipGetLastError());
}
```
